```python
import numpy as np

kernel_name = "gpu_gemm_a2a_m4096_k4096_n16384_e4m3e5m2_dequant"


ABSMAX_RTOL = 0.05

COMPUTE_DTYPE = "float32"
EPILOGUE = "dequant"
IN_QUANT = None
OUT_QUANT = None
IN_FP8 = "e4m3e5m2"
IN_DTYPES = {"x": "e4m3", "w_mat": "e5m2",
             "scale_x": "float32", "scale_w": "float32"}
WORLD_SIZE = 8

def _snap_fp8(x, mant_bits, max_abs, min_normal):
    a = np.abs(x.astype(np.float64))
    _m, _e = np.frexp(a)
    q = np.where(a >= min_normal, np.ldexp(1.0, _e - 1 - mant_bits),
                 min_normal * 2.0 ** -mant_bits)
    snapped = np.minimum(np.round(a / q) * q, max_abs)
    return (np.sign(x) * snapped).astype(np.float32)

def setup_inputs():

    x = _snap_fp8(np.random.randn(4096, 4096).astype(np.float32), 3, 448.0, 2.0 ** -6)
    w_mat = _snap_fp8(np.random.randn(4096, 16384).astype(np.float32), 2, 57344.0, 2.0 ** -14)
    scale_x = (0.02 * (1.0 + np.random.rand(1))).astype(np.float32)
    scale_w = (0.02 * (1.0 + np.random.rand(1))).astype(np.float32)
    return {"x": x, "w_mat": w_mat, "scale_x": scale_x, "scale_w": scale_w}

def reference(x, w_mat, scale_x, scale_w):
    acc = (x @ w_mat).astype(np.float32)
    y = (acc * (scale_x * scale_w)).astype(np.float32)
    return y
```

```cpp
#include <hip/hip_runtime.h>
#include <rccl/rccl.h>
#include <cstdio>

#define HIP_CHECK(expr) do { hipError_t _e = (expr); if (_e != hipSuccess) { \
    fprintf(stderr, "HIP error %s at %s:%d\n", hipGetErrorString(_e), __FILE__, __LINE__); \
    return; } } while (0)

namespace {

typedef int v8i __attribute__((ext_vector_type(8)));
typedef int v4i __attribute__((ext_vector_type(4)));
typedef int v2i __attribute__((ext_vector_type(2)));
typedef float v4f __attribute__((ext_vector_type(4)));
typedef unsigned u32x4 __attribute__((ext_vector_type(4)));
#define LAS __attribute__((address_space(3)))

constexpr int NR = 8;
constexpr int M = 4096, K = 4096, NW = 16384, NS = 2048, MS = 512;
constexpr int BM = 256, BN = 128, BK = 128, NT = K / BK;
constexpr int A_BYTES = BM * BK, B_BYTES = BK * BN, SLOT = A_BYTES + B_BYTES;
constexpr int NSLOT = 3;
constexpr int LDS_BYTES = NSLOT * SLOT + 2048;
constexpr int NWG = 256;
constexpr int NPROD = 128;

constexpr size_t WS_CTR = 0, WS_TICK = 128, WS_TMO = 256;
constexpr size_t WS_FLAG = 65536;
constexpr size_t WS_STAMP = 131072;
constexpr size_t WS_EXP = 1u << 20;
constexpr size_t SHARD = (size_t)MS * K;
constexpr unsigned long long DEADLINE_TICKS = 50000000ull;

#define A_AUX 0
#define PF_DIST 0
#define DO_ACQ 0

struct Args {
  const unsigned char* x; const unsigned char* W; const float* sx; const float* sw; float* out;
  unsigned char* ws[NR]; int rank;
};

#define BAR() __builtin_amdgcn_s_barrier()
#define SCHED() __builtin_amdgcn_sched_barrier(0)

__device__ __forceinline__ v8i mk8(v4i a, v4i b) { return __builtin_shufflevector(a, b, 0, 1, 2, 3, 4, 5, 6, 7); }

__global__ void __launch_bounds__(512, 2) fused_gemm(Args a) {
  __shared__ __attribute__((aligned(1024))) unsigned char smem[LDS_BYTES];
  LAS unsigned char* lds = (LAS unsigned char*)smem;
  const int tid = threadIdx.x, lane = tid & 63, wid = __builtin_amdgcn_readfirstlane(tid >> 6);
  const int b = blockIdx.x;
  const int xl = b & 7, half = (b >> 3) & 1, tn = b >> 4;
  const int src = (a.rank + xl) & 7;
  const int tm = src * 2 + half;
  unsigned char* const myws = a.ws[a.rank];
  LAS unsigned* ctl = (LAS unsigned*)(lds + NSLOT * SLOT);

  unsigned call = __hip_atomic_load((unsigned*)(myws + WS_CTR), __ATOMIC_RELAXED, __HIP_MEMORY_SCOPE_AGENT) + 1u;
  call = __builtin_amdgcn_readfirstlane(call);
  const unsigned par = call & 1u;
  if (b < NPROD) {
    const int h = b >> 6, r0 = (b & 63) * 4;
    __amdgpu_buffer_rsrc_t rs = __builtin_amdgcn_make_buffer_rsrc((void*)(a.x + (size_t)(h * 256 + r0) * K), 0, 4 * K, 0x00020000);
    __amdgpu_buffer_rsrc_t rd = __builtin_amdgcn_make_buffer_rsrc((void*)(myws + WS_EXP + par * SHARD + (size_t)h * (SHARD / 2)), 0, (unsigned)(SHARD / 2), 0x00020000);
    u32x4 v0 = __builtin_amdgcn_raw_buffer_load_b128(rs, tid * 16, 0, 0);
    u32x4 v1 = __builtin_amdgcn_raw_buffer_load_b128(rs, tid * 16 + 8192, 0, 0);
    const int o = tid * 16;
    const int row = r0 + (o >> 12), kt = (o & 4095) >> 7, c = o & 127;
    __builtin_amdgcn_raw_buffer_store_b128(v0, rd, kt * A_BYTES + row * BK + c, 0, 17);
    __builtin_amdgcn_raw_buffer_store_b128(v1, rd, kt * A_BYTES + (row + 2) * BK + c, 0, 17);
    asm volatile("s_waitcnt vmcnt(0)" ::: "memory");
    __syncthreads();
    if (tid < NR && tid != a.rank) {
      unsigned* f = (unsigned*)(a.ws[tid] + WS_FLAG) + a.rank * NPROD + b;
      __hip_atomic_store(f, call, __ATOMIC_RELAXED, __HIP_MEMORY_SCOPE_SYSTEM);
    }
  }
  unsigned ticket = 0;

  if (wid < 4) {
    const int wm = wid >> 1, wn = wid & 1;
    const int r = lane & 15, g = lane >> 4;
    const unsigned aoff0 = (wm * 128 + r) * 128 + ((g ^ ((r >> 1) & 7)) * 16);
    const unsigned aoff1 = aoff0 ^ 64;
    const int q = r >> 1, p = r & 1;
    unsigned boff[4];
#pragma unroll
    for (int nn = 0; nn < 4; ++nn) boff[nn] = A_BYTES + 2 * g * 1024 + q * 128 + (((wn * 4 + nn) ^ ((q >> 1) | ((g & 1) << 2))) * 16) + p * 8;
    v4f acc[8][4];
#pragma unroll
    for (int m = 0; m < 8; ++m)
#pragma unroll
      for (int nn = 0; nn < 4; ++nn) acc[m][nn] = (v4f){0.f, 0.f, 0.f, 0.f};
    v8i Bf[2][4];
    v8i Af[4];
#define RDA(slotoff, m) mk8(*(const LAS v4i*)(lds + (slotoff) + aoff0 + (m) * 2048), *(const LAS v4i*)(lds + (slotoff) + aoff1 + (m) * 2048))
#define RDB2(slotoff, nn, r_) __builtin_amdgcn_ds_read_tr8_b64_v2i32((LAS v2i*)(lds + (slotoff) + boff[nn] + ((r_) & 1) * 1024 + ((r_) >> 1) * 8192))
#define RDB(slotoff, nn) ({ v2i b0 = RDB2(slotoff, nn, 0), b1 = RDB2(slotoff, nn, 1), b2 = RDB2(slotoff, nn, 2), b3 = RDB2(slotoff, nn, 3); \
      v4i lo = __builtin_shufflevector(b0, b1, 0, 1, 2, 3), hi = __builtin_shufflevector(b2, b3, 0, 1, 2, 3); mk8(lo, hi); })
#define MMA4(m, Bc, Ac) do { _Pragma("unroll") for (int nn = 0; nn < 4; ++nn) \
      acc[m][nn] = __builtin_amdgcn_mfma_scale_f32_16x16x128_f8f6f4(Bc[nn], Ac, acc[m][nn], 1, 0, 0, 0, 0, 0); } while (0)
    BAR();
    BAR();
    {
      const unsigned so = 0;
#pragma unroll
      for (int nn = 0; nn < 4; ++nn) Bf[0][nn] = RDB(so, nn);
      Af[0] = RDA(so, 0); Af[1] = RDA(so, 1);
    }
    unsigned so = 0;
#define KSTEP(cur, nxt, LAST) do { \
      const unsigned sn = (so == 2 * SLOT) ? 0u : so + SLOT; \
      Af[2] = RDA(so, 2); SCHED(); MMA4(0, Bf[cur], Af[0]); SCHED(); \
      Af[3] = RDA(so, 3); SCHED(); MMA4(1, Bf[cur], Af[1]); SCHED(); \
      Af[0] = RDA(so, 4); SCHED(); MMA4(2, Bf[cur], Af[2]); SCHED(); \
      Af[1] = RDA(so, 5); SCHED(); MMA4(3, Bf[cur], Af[3]); SCHED(); \
      Af[2] = RDA(so, 6); Af[3] = RDA(so, 7); SCHED(); MMA4(4, Bf[cur], Af[0]); SCHED(); \
      MMA4(5, Bf[cur], Af[1]); SCHED(); \
      asm volatile("s_waitcnt lgkmcnt(0)" ::: "memory"); \
      BAR(); SCHED(); \
      if (!(LAST)) { Bf[nxt][0] = RDB(sn, 0); Bf[nxt][1] = RDB(sn, 1); Af[0] = RDA(sn, 0); } SCHED(); \
      MMA4(6, Bf[cur], Af[2]); SCHED(); \
      if (!(LAST)) { Bf[nxt][2] = RDB(sn, 2); Bf[nxt][3] = RDB(sn, 3); Af[1] = RDA(sn, 1); } SCHED(); \
      MMA4(7, Bf[cur], Af[3]); SCHED(); \
      so = sn; } while (0)
    for (int kt = 0; kt < NT - 2; kt += 2) {
      KSTEP(0, 1, false);
      KSTEP(1, 0, false);
    }
    KSTEP(0, 1, false);
    KSTEP(1, 0, true);
    float s = a.sx[0] * a.sw[0];
    if (ctl[1]) s = __builtin_nanf("");
#pragma unroll
    for (int m = 0; m < 8; ++m)
#pragma unroll
      for (int nn = 0; nn < 4; ++nn) {
        const int row = wm * 128 + m * 16 + r;
        const int c = wn * 16 + nn * 4 + g;
        v4f v = acc[m][nn] * s;
        *(LAS v4f*)(lds + row * 512 + ((c ^ (row & 7)) * 16)) = v;
      }
    asm volatile("s_waitcnt lgkmcnt(0)" ::: "memory");
    BAR();
  } else {
    const int hw = wid - 4;
    const bool local = (src == a.rank);
    const unsigned char* abase = local ? (a.x + (size_t)half * BM * K) : (a.ws[src] + WS_EXP + par * SHARD + (size_t)half * (SHARD / 2));
    const unsigned rowstride = local ? K : BK;
    const unsigned kstride = local ? BK : A_BYTES;
    __amdgpu_buffer_rsrc_t rsA = __builtin_amdgcn_make_buffer_rsrc((void*)abase, 0, BM * K, 0x00020000);
    __amdgpu_buffer_rsrc_t rsB = __builtin_amdgcn_make_buffer_rsrc((void*)(a.W + a.rank * NS + tn * BN), 0, (unsigned)((size_t)K * NW - (a.rank * NS + tn * BN)), 0x00020000);
    const int lq = lane >> 3, lc = lane & 7;
    const unsigned vA0 = lq * rowstride + ((lc ^ (lane >> 4)) * 16);
    const unsigned vA1 = lq * rowstride + ((lc ^ (4 + (lane >> 4))) * 16);
    const unsigned vB0 = lq * NW + ((lc ^ (lq >> 1)) * 16);
    const unsigned vB1 = lq * NW + ((lc ^ ((lq >> 1) | 4)) * 16);
#define DMA_A(slotoff, i, kt) __builtin_amdgcn_raw_ptr_buffer_load_lds(rsA, (LAS void*)(lds + (slotoff) + (i) * 1024), 16, ((i) & 1) ? vA1 : vA0, (i) * 8 * rowstride + (kt) * kstride, 0, A_AUX)
#define DMA_B(slotoff, j, kt) __builtin_amdgcn_raw_ptr_buffer_load_lds(rsB, (LAS void*)(lds + (slotoff) + A_BYTES + (j) * 1024), 16, (((j) >> 1) & 1) ? vB1 : vB0, ((kt) * BK + (j) * 8) * NW, 0, 0)
#define ISSUE_B(kt) do { const unsigned so_ = ((kt) % NSLOT) * SLOT; _Pragma("unroll") for (int j = 0; j < 16; ++j) DMA_B(so_, j, kt); } while (0)
#define ISSUE_A(kt) do { const unsigned so_ = ((kt) % NSLOT) * SLOT; \
      if (hw == 1) { _Pragma("unroll") for (int i = 0; i < 16; ++i) DMA_A(so_, i, kt); } \
      else { _Pragma("unroll") for (int i = 16; i < 32; ++i) DMA_A(so_, i, kt); } } while (0)
    if (hw == 3) { ISSUE_B(0); ISSUE_B(1); ISSUE_B(2); }
    if (hw == 0) {
      unsigned bad = 0;
      if (!local) {
        const unsigned* f = (const unsigned*)(myws + WS_FLAG) + src * NPROD + half * 64 + lane;
        unsigned* tmo = (unsigned*)(myws + WS_TMO);
        const unsigned long long t0 = __builtin_amdgcn_s_memrealtime();
        for (;;) {
          const unsigned v = __hip_atomic_load(f, __ATOMIC_RELAXED, __HIP_MEMORY_SCOPE_SYSTEM);
          if (__all(v >= call)) break;
          if (__hip_atomic_load(tmo, __ATOMIC_RELAXED, __HIP_MEMORY_SCOPE_AGENT) == call || __builtin_amdgcn_s_memrealtime() - t0 > DEADLINE_TICKS) {
            if (lane == 0) __hip_atomic_store(tmo, call, __ATOMIC_RELAXED, __HIP_MEMORY_SCOPE_AGENT);
            break;
          }
          __builtin_amdgcn_s_sleep(1);
        }
        bad = (__hip_atomic_load(tmo, __ATOMIC_RELAXED, __HIP_MEMORY_SCOPE_AGENT) == call);
      }
      if (lane == 0) ctl[1] = bad;
      asm volatile("s_waitcnt lgkmcnt(0)" ::: "memory");
    }
    BAR();
    if (tid == 256) ticket = __hip_atomic_fetch_add((unsigned*)(myws + WS_TICK), 1u, __ATOMIC_RELAXED, __HIP_MEMORY_SCOPE_AGENT);
    const unsigned vP = tn * 2048 + lane * 16;
#define PREFETCH(kt) do { \
      __builtin_amdgcn_raw_ptr_buffer_load_lds(rsA, (LAS void*)(lds + NSLOT * SLOT + 1024), 16, vP, (kt) * A_BYTES, 0, 0); \
      __builtin_amdgcn_raw_ptr_buffer_load_lds(rsA, (LAS void*)(lds + NSLOT * SLOT + 1024), 16, vP, (kt) * A_BYTES + 1024, 0, 0); } while (0)
    if (PF_DIST > 0 && hw == 0 && !local) { for (int kt = 0; kt < PF_DIST; ++kt) PREFETCH(kt); }
    if (hw == 1 || hw == 2) { ISSUE_A(0); ISSUE_A(1); ISSUE_A(2); }
    if (hw != 0) asm volatile("s_waitcnt vmcnt(32)" ::: "memory");
    BAR();
    for (int kt = 0; kt < NT; ++kt) {
      if (hw != 0) {
        if (kt + 2 < NT) asm volatile("s_waitcnt vmcnt(16)" ::: "memory");
        else asm volatile("s_waitcnt vmcnt(0)" ::: "memory");
      }
      BAR();
      if (kt + 3 < NT) { if (hw == 3) ISSUE_B(kt + 3); else if (hw != 0) ISSUE_A(kt + 3); }
      if (PF_DIST > 0 && hw == 0 && !local && kt + PF_DIST < NT) PREFETCH(kt + PF_DIST);
    }
    BAR();
  }
  {
    __amdgpu_buffer_rsrc_t rsC = __builtin_amdgcn_make_buffer_rsrc((void*)(a.out + (size_t)tm * BM * NS + tn * BN), 0, BM * NS * 4, 0x00020000);
#pragma unroll
    for (int i = 0; i < 16; ++i) {
      const int row = wid * 32 + i * 2 + (lane >> 5), c = lane & 31;
      v4f v = *(const LAS v4f*)(lds + row * 512 + ((c ^ (row & 7)) * 16));
      __builtin_amdgcn_raw_buffer_store_b128(__builtin_bit_cast(u32x4, v), rsC, row * NS * 4 + c * 16, 0, 16);
    }
  }
  if (tid == 256 && (ticket % NWG) == NWG - 1)
    __hip_atomic_store((unsigned*)(myws + WS_CTR), call, __ATOMIC_RELAXED, __HIP_MEMORY_SCOPE_AGENT);
}

}

extern "C" void kernel_launch(int rank, int nranks, ncclComm_t comm,
                              hipStream_t stream,
                              void* const* d_in, const int* in_sizes, int n_in,
                              void* d_out, int out_size,
                              const int* in_dtypes, int out_dtype,
                              void* const* d_out_all,
                              void* const* d_ws_all, size_t ws_bytes) {
  if (nranks != NR || n_in < 4) { fprintf(stderr, "unsupported configuration\n"); return; }
  Args a;
  a.x = (const unsigned char*)d_in[0];
  a.W = (const unsigned char*)d_in[1];
  a.sx = (const float*)d_in[2];
  a.sw = (const float*)d_in[3];
  a.out = (float*)d_out;
  for (int p = 0; p < NR; ++p) a.ws[p] = (unsigned char*)d_ws_all[p];
  a.rank = rank;
  hipLaunchKernelGGL(fused_gemm, dim3(NWG), dim3(512), 0, stream, a);
  HIP_CHECK(hipGetLastError());
}
```

```cpp
#include <hip/hip_runtime.h>
#include <rccl/rccl.h>
#include <cstdio>

#define HIP_CHECK(expr) do { hipError_t _e = (expr); if (_e != hipSuccess) { \
    fprintf(stderr, "HIP error %s at %s:%d\n", hipGetErrorString(_e), __FILE__, __LINE__); \
    return; } } while (0)

namespace {

typedef int v8i __attribute__((ext_vector_type(8)));
typedef int v4i __attribute__((ext_vector_type(4)));
typedef int v2i __attribute__((ext_vector_type(2)));
typedef float v4f __attribute__((ext_vector_type(4)));
typedef unsigned u32x4 __attribute__((ext_vector_type(4)));
#define LAS __attribute__((address_space(3)))

constexpr int NR = 8;
constexpr int M = 4096, K = 4096, NW = 16384, NS = 2048, MS = 512;
constexpr int BM = 256, BN = 128, BK = 128, NT = K / BK;
constexpr int A_BYTES = BM * BK, B_BYTES = BK * BN, SLOT = A_BYTES + B_BYTES;
constexpr int NSLOT = 3;
constexpr int LDS_BYTES = NSLOT * SLOT + 2048;
constexpr int NWG = 256;
#define NPUSHV 2
#define NEARLY 1
#define PBASE 6
constexpr int NPUSH = NPUSHV;
constexpr int PSTR = NPUSH <= 3 ? 8 : 6;

constexpr size_t WS_CTR = 0, WS_TICK = 128, WS_TMO = 256;
constexpr size_t WS_FLAGE = 65536;
constexpr size_t WS_FLAGP = 65536 + 16384;
constexpr size_t WS_STAMP = 131072;
constexpr size_t WS_EXP = 1u << 20;
constexpr size_t WS_STG = 8u << 20;
constexpr size_t SHARD = (size_t)MS * K;
constexpr unsigned STG_SRC = 2 * 4 * A_BYTES;
constexpr unsigned long long DEADLINE_TICKS = 50000000ull;

struct Args {
  const unsigned char* x; const unsigned char* W; const float* sx; const float* sw; float* out;
  unsigned char* wsr[NR];
  int rank;
};

#define BAR() __builtin_amdgcn_s_barrier()
#define SCHED() __builtin_amdgcn_sched_barrier(0)

__device__ __forceinline__ v8i mk8(v4i a, v4i b) { return __builtin_shufflevector(a, b, 0, 1, 2, 3, 4, 5, 6, 7); }

__global__ void __launch_bounds__(512, 2) fused_gemm(Args a) {
  __shared__ __attribute__((aligned(1024))) unsigned char smem[LDS_BYTES];
  LAS unsigned char* lds = (LAS unsigned char*)smem;
  const int tid = threadIdx.x, lane = tid & 63, wid = __builtin_amdgcn_readfirstlane(tid >> 6);
  const int b = blockIdx.x;
  const int xl = b & 7, half = (b >> 3) & 1, tn = b >> 4;
  const int src = (a.rank + xl) & 7;
  const int tm = src * 2 + half;
  unsigned char* const myws = a.wsr[0];
  LAS unsigned* ctl = (LAS unsigned*)(lds + NSLOT * SLOT);

  const unsigned craw = __hip_atomic_load((unsigned*)(myws + WS_CTR), __ATOMIC_RELAXED, __HIP_MEMORY_SCOPE_AGENT);
  unsigned call;
  const bool exporter = (xl < 4);
  const int rw = (b >> 3) * 4 + (b & 3);
  unsigned char* ppeer = nullptr; unsigned* pflag = nullptr;
  if (exporter) {
    const int h = rw >> 6, r0 = (rw & 63) * 4;
    const __amdgpu_buffer_rsrc_t rs = __builtin_amdgcn_make_buffer_rsrc((void*)(a.x + (size_t)(h * 256 + r0) * K), 0, 4 * K, 0x00020000);
    u32x4 v0 = __builtin_amdgcn_raw_buffer_load_b128(rs, tid * 16, 0, 0);
    u32x4 v1 = __builtin_amdgcn_raw_buffer_load_b128(rs, tid * 16 + 8192, 0, 0);
    unsigned char* pl = a.wsr[lane & 7];
    call = __builtin_amdgcn_readfirstlane(craw) + 1u;
    const __amdgpu_buffer_rsrc_t rd = __builtin_amdgcn_make_buffer_rsrc((void*)(myws + WS_EXP + (call & 1u) * SHARD + (size_t)h * (SHARD / 2)), 0, (unsigned)(SHARD / 2), 0x00020000);
    const int o = tid * 16;
    const int row = r0 + (o >> 12), kt = (o & 4095) >> 7, c = o & 127;
    __builtin_amdgcn_raw_buffer_store_b128(v0, rd, kt * A_BYTES + row * BK + c, 0, 17);
    __builtin_amdgcn_raw_buffer_store_b128(v1, rd, kt * A_BYTES + (row + 2) * BK + c, 0, 17);
    asm volatile("s_waitcnt vmcnt(0)" ::: "memory");
    BAR();
    if (wid == 4 && lane < NR && lane != 0)
      __hip_atomic_store((unsigned*)(pl + WS_FLAGE) + a.rank * 128 + rw, call, __ATOMIC_RELAXED, __HIP_MEMORY_SCOPE_SYSTEM);
  } else if (wid < 4 && NPUSH > 0) {
    const int u = rw * 4 + wid;
    if (u < 64 * 7) {
      const int pc = u / 7, po = 1 + u % 7, ph = pc >> 5, pi = pc & 31;
      const __amdgpu_buffer_rsrc_t rsX = __builtin_amdgcn_make_buffer_rsrc((void*)(a.x + (size_t)(ph * 256 + pi * 8) * K), 0, 8 * K, 0x00020000);
      const unsigned vX = (lane >> 3) * K + (lane & 7) * 16;
      u32x4 v[NPUSH > 0 ? NPUSH : 1];
#pragma unroll
      for (int k = 0; k < NPUSH; ++k) v[k] = __builtin_amdgcn_raw_buffer_load_b128(rsX, vX, k * BK, 0);
      ppeer = a.wsr[po];
      call = __builtin_amdgcn_readfirstlane(craw) + 1u;
      const size_t so = WS_STG + (size_t)((call & 1u) * NR + a.rank) * STG_SRC + (size_t)(ph * 4) * A_BYTES + pi * 1024;
      const __amdgpu_buffer_rsrc_t rd = __builtin_amdgcn_make_buffer_rsrc((void*)(ppeer + so), 0, 4 * A_BYTES, 0x00020000);
#pragma unroll
      for (int k = 0; k < NPUSH; ++k) __builtin_amdgcn_raw_buffer_store_b128(v[k], rd, lane * 16, k * A_BYTES, 17);
      pflag = (unsigned*)(ppeer + WS_FLAGP) + a.rank * 64 + pc;
    }
  }
  call = __builtin_amdgcn_readfirstlane(craw) + 1u;
  const unsigned par = call & 1u;
  unsigned ticket = 0;

  if (wid < 4) {
    const int wm = wid >> 1, wn = wid & 1;
    const int r = lane & 15, g = lane >> 4;
    const unsigned aoff0 = (wm * 128 + r) * 128 + ((g ^ ((r >> 1) & 7)) * 16);
    const unsigned aoff1 = aoff0 ^ 64;
    const int q = r >> 1, p = r & 1;
    unsigned boff[4];
#pragma unroll
    for (int nn = 0; nn < 4; ++nn) boff[nn] = A_BYTES + 2 * g * 1024 + q * 128 + (((wn * 4 + nn) ^ ((q >> 1) | ((g & 1) << 2))) * 16) + p * 8;
    v4f acc[8][4];
#pragma unroll
    for (int m = 0; m < 8; ++m)
#pragma unroll
      for (int nn = 0; nn < 4; ++nn) acc[m][nn] = (v4f){0.f, 0.f, 0.f, 0.f};
    v8i Bf[2][4];
    v8i Af[4];
#define RDA(slotoff, m) mk8(*(const LAS v4i*)(lds + (slotoff) + aoff0 + (m) * 2048), *(const LAS v4i*)(lds + (slotoff) + aoff1 + (m) * 2048))
#define RDB2(slotoff, nn, r_) __builtin_amdgcn_ds_read_tr8_b64_v2i32((LAS v2i*)(lds + (slotoff) + boff[nn] + ((r_) & 1) * 1024 + ((r_) >> 1) * 8192))
#define RDB(slotoff, nn) ({ v2i b0 = RDB2(slotoff, nn, 0), b1 = RDB2(slotoff, nn, 1), b2 = RDB2(slotoff, nn, 2), b3 = RDB2(slotoff, nn, 3); \
      v4i lo = __builtin_shufflevector(b0, b1, 0, 1, 2, 3), hi = __builtin_shufflevector(b2, b3, 0, 1, 2, 3); mk8(lo, hi); })
#define MMA4(m, Bc, Ac) do { _Pragma("unroll") for (int nn = 0; nn < 4; ++nn) \
      acc[m][nn] = __builtin_amdgcn_mfma_scale_f32_16x16x128_f8f6f4(Bc[nn], Ac, acc[m][nn], 1, 0, 0, 0, 0, 0); } while (0)
    BAR();
    if (pflag) {
      asm volatile("s_waitcnt vmcnt(0)" ::: "memory");
      if (lane == 0) __hip_atomic_store(pflag, call, __ATOMIC_RELAXED, __HIP_MEMORY_SCOPE_SYSTEM);
    }
    BAR();
    {
      const unsigned so = 0;
#pragma unroll
      for (int nn = 0; nn < 4; ++nn) Bf[0][nn] = RDB(so, nn);
      Af[0] = RDA(so, 0); Af[1] = RDA(so, 1);
    }
    unsigned so = 0;
#define KSTEP(cur, nxt, LAST) do { \
      const unsigned sn = (so == 2 * SLOT) ? 0u : so + SLOT; \
      Af[2] = RDA(so, 2); SCHED(); MMA4(0, Bf[cur], Af[0]); SCHED(); \
      Af[3] = RDA(so, 3); SCHED(); MMA4(1, Bf[cur], Af[1]); SCHED(); \
      Af[0] = RDA(so, 4); SCHED(); MMA4(2, Bf[cur], Af[2]); SCHED(); \
      Af[1] = RDA(so, 5); SCHED(); MMA4(3, Bf[cur], Af[3]); SCHED(); \
      Af[2] = RDA(so, 6); Af[3] = RDA(so, 7); SCHED(); MMA4(4, Bf[cur], Af[0]); SCHED(); \
      MMA4(5, Bf[cur], Af[1]); SCHED(); \
      asm volatile("s_waitcnt lgkmcnt(0)" ::: "memory"); \
      BAR(); SCHED(); \
      if (!(LAST)) { Bf[nxt][0] = RDB(sn, 0); Bf[nxt][1] = RDB(sn, 1); Af[0] = RDA(sn, 0); } SCHED(); \
      MMA4(6, Bf[cur], Af[2]); SCHED(); \
      if (!(LAST)) { Bf[nxt][2] = RDB(sn, 2); Bf[nxt][3] = RDB(sn, 3); Af[1] = RDA(sn, 1); } SCHED(); \
      MMA4(7, Bf[cur], Af[3]); SCHED(); \
      so = sn; } while (0)
    for (int kt = 0; kt < NT - 2; kt += 2) {
      KSTEP(0, 1, false);
      KSTEP(1, 0, false);
    }
    KSTEP(0, 1, false);
    float s = a.sx[0] * a.sw[0];
    if (ctl[1]) s = __builtin_nanf("");
    const __amdgpu_buffer_rsrc_t rsC = __builtin_amdgcn_make_buffer_rsrc((void*)(a.out + (size_t)(tm * BM + wm * 128) * NS + tn * BN + wn * 64), 0, 128 * NS * 4, 0x00020000);
    const unsigned vC = (r & 3) * (NS * 4) + (r >> 2) * 64 + g * 16;
#define DPPF(old, src, ctrl, bank) __builtin_bit_cast(float, __builtin_amdgcn_update_dpp(__builtin_bit_cast(int, (float)(old)), __builtin_bit_cast(int, (float)(src)), ctrl, 0xf, bank, false))
#define OUTM(m) do { _Pragma("unroll") for (int e = 0; e < 4; ++e) { \
      const float r0 = acc[m][0][e] * s, r1 = acc[m][1][e] * s, r2 = acc[m][2][e] * s, r3 = acc[m][3][e] * s; \
      const float s0 = DPPF(r0, r2, 0x128, 0xc), s2 = DPPF(r2, r0, 0x128, 0x3); \
      const float s1 = DPPF(r1, r3, 0x128, 0xc), s3 = DPPF(r3, r1, 0x128, 0x3); \
      acc[m][0][e] = DPPF(s0, s1, 0x114, 0xa); acc[m][1][e] = DPPF(s1, s0, 0x104, 0x5); \
      acc[m][2][e] = DPPF(s2, s3, 0x114, 0xa); acc[m][3][e] = DPPF(s3, s2, 0x104, 0x5); } \
      _Pragma("unroll") for (int t = 0; t < 4; ++t) \
        __builtin_amdgcn_raw_buffer_store_b128(__builtin_bit_cast(u32x4, acc[m][t]), rsC, vC, ((m) * 16 + t * 4) * (NS * 4), 16); \
      SCHED(); } while (0)
    Af[2] = RDA(so, 2); SCHED(); MMA4(0, Bf[1], Af[0]); SCHED();
    Af[3] = RDA(so, 3); SCHED(); MMA4(1, Bf[1], Af[1]); SCHED();
    Af[0] = RDA(so, 4); SCHED(); MMA4(2, Bf[1], Af[2]); SCHED(); OUTM(0);
    Af[1] = RDA(so, 5); SCHED(); MMA4(3, Bf[1], Af[3]); SCHED(); OUTM(1);
    Af[2] = RDA(so, 6); Af[3] = RDA(so, 7); SCHED(); MMA4(4, Bf[1], Af[0]); SCHED(); OUTM(2);
    MMA4(5, Bf[1], Af[1]); SCHED(); OUTM(3);
    MMA4(6, Bf[1], Af[2]); SCHED(); OUTM(4);
    MMA4(7, Bf[1], Af[3]); SCHED(); OUTM(5);
    OUTM(6);
    OUTM(7);
  } else {
    const int hw = wid - 4;
    const bool local = (src == a.rank);
    const unsigned char* abase = local ? (a.x + (size_t)half * BM * K) : (a.wsr[xl] + WS_EXP + par * SHARD + (size_t)half * (SHARD / 2));
    const unsigned rowstride = local ? K : BK;
    const unsigned kstride = local ? BK : A_BYTES;
    const __amdgpu_buffer_rsrc_t rsA = __builtin_amdgcn_make_buffer_rsrc((void*)abase, 0, BM * K, 0x00020000);
    const __amdgpu_buffer_rsrc_t rsS = local ? rsA : __builtin_amdgcn_make_buffer_rsrc((void*)(myws + WS_STG + (size_t)(par * NR + src) * STG_SRC + (size_t)half * 4 * A_BYTES), 0, 4 * A_BYTES, 0x00020000);
    const __amdgpu_buffer_rsrc_t rsB = __builtin_amdgcn_make_buffer_rsrc((void*)(a.W + a.rank * NS + tn * BN), 0, (unsigned)((size_t)K * NW - (a.rank * NS + tn * BN)), 0x00020000);
    const int lq = lane >> 3, lc = lane & 7;
    const unsigned vA0 = lq * rowstride + ((lc ^ (lane >> 4)) * 16);
    const unsigned vA1 = lq * rowstride + ((lc ^ (4 + (lane >> 4))) * 16);
    const unsigned vB0 = lq * NW + ((lc ^ (lq >> 1)) * 16);
    const unsigned vB1 = lq * NW + ((lc ^ ((lq >> 1) | 4)) * 16);
    const int pos0 = PBASE + (tn & 7);
    auto kstep_of = [&](int j) -> int {
      int kt = j + NPUSH;
#pragma unroll
      for (int i = 0; i < NPUSH; ++i) { const int p = pos0 + i * PSTR; if (j > p) kt -= 1; }
#pragma unroll
      for (int i = 0; i < NPUSH; ++i) { const int p = pos0 + i * PSTR; if (j == p) kt = i; }
      return kt;
    };
#define DMA_A(rs, slotoff, i, kt) __builtin_amdgcn_raw_ptr_buffer_load_lds(rs, (LAS void*)(lds + (slotoff) + (i) * 1024), 16, ((i) & 1) ? vA1 : vA0, (i) * 8 * rowstride + (kt) * kstride, 0, 0)
#define DMA_B(slotoff, j, kt) __builtin_amdgcn_raw_ptr_buffer_load_lds(rsB, (LAS void*)(lds + (slotoff) + A_BYTES + (j) * 1024), 16, (((j) >> 1) & 1) ? vB1 : vB0, ((kt) * BK + (j) * 8) * NW, 0, 0)
#define ISSUE_B(js) do { const unsigned so_ = ((js) % NSLOT) * SLOT; const int kt_ = kstep_of(js); _Pragma("unroll") for (int j = 0; j < 16; ++j) DMA_B(so_, j, kt_); } while (0)
#define ISSUE_A(js) do { const unsigned so_ = ((js) % NSLOT) * SLOT; const int kt_ = kstep_of(js); const __amdgpu_buffer_rsrc_t r_ = (kt_ < NPUSH) ? rsS : rsA; \
      if (hw == 1) { _Pragma("unroll") for (int i = 0; i < 16; ++i) DMA_A(r_, so_, i, kt_); } \
      else { _Pragma("unroll") for (int i = 16; i < 32; ++i) DMA_A(r_, so_, i, kt_); } } while (0)
    if (hw == 3) { ISSUE_B(0); ISSUE_B(1); ISSUE_B(2); }
    unsigned* const tmo = (unsigned*)(myws + WS_TMO);
    const unsigned long long t0 = __builtin_amdgcn_s_memrealtime();
#define WAIT_FLAGS(f, act) do { for (;;) { \
          const unsigned v_ = __hip_atomic_load(f, __ATOMIC_RELAXED, __HIP_MEMORY_SCOPE_SYSTEM); \
          const unsigned tv_ = __hip_atomic_load(tmo, __ATOMIC_RELAXED, __HIP_MEMORY_SCOPE_AGENT); \
          asm volatile("" :: "v"(tv_), "v"(v_));     \
          if (__all(v_ >= call)) break; \
          if (tv_ == call || __builtin_amdgcn_s_memrealtime() - t0 > DEADLINE_TICKS) { \
            if (lane == 0) __hip_atomic_store(tmo, call, __ATOMIC_RELAXED, __HIP_MEMORY_SCOPE_AGENT); \
            asm volatile("s_waitcnt vmcnt(0)" ::: "memory"); \
            break; \
          } } } while (0)
    const unsigned* fE = (const unsigned*)(myws + WS_FLAGE) + src * 128 + half * 64 + lane;
    const unsigned* fP = (const unsigned*)(myws + WS_FLAGP) + src * 64 + half * 32 + (lane & 31);
    if (hw == 0 && !local) {
      WAIT_FLAGS(fE, true);
    }
    BAR();
    if (tid == 256) ticket = __hip_atomic_fetch_add((unsigned*)(myws + WS_TICK), 1u, __ATOMIC_RELAXED, __HIP_MEMORY_SCOPE_AGENT);
    if (hw == 1 || hw == 2) { ISSUE_A(0); ISSUE_A(1); ISSUE_A(2); }
    if (hw != 0) asm volatile("s_waitcnt vmcnt(32)" ::: "memory");
    BAR();
    bool pseen = local || NPUSH == 0;
    for (int js = 0; js < NT - 1; ++js) {
      if (hw != 0) {
        if (js + 2 < NT) asm volatile("s_waitcnt vmcnt(16)" ::: "memory");
        else asm volatile("s_waitcnt vmcnt(0)" ::: "memory");
      } else if (!pseen) {
        if (js + 3 >= pos0) { WAIT_FLAGS(fP, actP); pseen = true; }
        else {
          const unsigned v_ = __hip_atomic_load(fP, __ATOMIC_RELAXED, __HIP_MEMORY_SCOPE_SYSTEM);
          pseen = __all(v_ >= call);
        }
      }
      if (hw == 0 && js == pos0 - 3) {
        const unsigned tv2 = __hip_atomic_load(tmo, __ATOMIC_RELAXED, __HIP_MEMORY_SCOPE_AGENT);
        if (lane == 0) ctl[1] = (tv2 == call);
        asm volatile("s_waitcnt lgkmcnt(0)" ::: "memory");
      }
      BAR();
      if (js + 3 < NT) { if (hw == 3) ISSUE_B(js + 3); else if (hw != 0) ISSUE_A(js + 3); }
    }
  }
  if (tid == 256 && (ticket % NWG) == NWG - 1)
    __hip_atomic_store((unsigned*)(myws + WS_CTR), call, __ATOMIC_RELAXED, __HIP_MEMORY_SCOPE_AGENT);
}

}

extern "C" void kernel_launch(int rank, int nranks, ncclComm_t comm,
                              hipStream_t stream,
                              void* const* d_in, const int* in_sizes, int n_in,
                              void* d_out, int out_size,
                              const int* in_dtypes, int out_dtype,
                              void* const* d_out_all,
                              void* const* d_ws_all, size_t ws_bytes) {
  if (nranks != NR || n_in < 4) { fprintf(stderr, "unsupported configuration\n"); return; }
  Args a;
  a.x = (const unsigned char*)d_in[0];
  a.W = (const unsigned char*)d_in[1];
  a.sx = (const float*)d_in[2];
  a.sw = (const float*)d_in[3];
  a.out = (float*)d_out;
  for (int p = 0; p < NR; ++p) a.wsr[p] = (unsigned char*)d_ws_all[(rank + p) & 7];
  a.rank = rank;
  hipLaunchKernelGGL(fused_gemm, dim3(NWG), dim3(512), 0, stream, a);
  HIP_CHECK(hipGetLastError());
}
```

```cpp
#include <hip/hip_runtime.h>
#include <rccl/rccl.h>
#include <cstdio>

#define HIP_CHECK(expr) do { hipError_t _e = (expr); if (_e != hipSuccess) { \
    fprintf(stderr, "HIP error %s at %s:%d\n", hipGetErrorString(_e), __FILE__, __LINE__); \
    return; } } while (0)

namespace {

typedef int v8i __attribute__((ext_vector_type(8)));
typedef int v4i __attribute__((ext_vector_type(4)));
typedef int v2i __attribute__((ext_vector_type(2)));
typedef float v4f __attribute__((ext_vector_type(4)));
typedef unsigned u32x4 __attribute__((ext_vector_type(4)));
#define LAS __attribute__((address_space(3)))

constexpr int NR = 8;
constexpr int M = 4096, K = 4096, NW = 16384, NS = 2048, MS = 512;
constexpr int BM = 256, BN = 128, BK = 128, NT = K / BK;
constexpr int A_BYTES = BM * BK, B_BYTES = BK * BN, SLOT = A_BYTES + B_BYTES;
constexpr int NSLOT = 3;
constexpr int LDS_BYTES = NSLOT * SLOT + 2048 + 2048;
constexpr int NWG = 256;
#define NPUSHV 2
#define NEARLY 1
#define PBASE 6
#define ROT 0
#define DE 0
#define DL 0
#define HOLD1 0
constexpr int NPUSH = NPUSHV;
constexpr int PSTR = NPUSH <= 3 ? 8 : 6;

constexpr size_t WS_CTR = 0, WS_TICK = 128, WS_TMO = 256;
constexpr size_t WS_FLAGE = 65536;
constexpr size_t WS_FLAGP = 65536 + 16384;
constexpr size_t WS_STAMP = 131072;
constexpr size_t WS_EXP = 1u << 20;
constexpr size_t WS_STG = 8u << 20;
constexpr size_t SHARD = (size_t)MS * K;
constexpr unsigned STG_SRC = 2 * 4 * A_BYTES;
constexpr unsigned long long DEADLINE_TICKS = 50000000ull;

struct Args {
  const unsigned char* x; const unsigned char* W; const float* sx; const float* sw; float* out;
  unsigned char* wsr[NR];
  int rank;
};

#define BAR() __builtin_amdgcn_s_barrier()
#define SCHED() __builtin_amdgcn_sched_barrier(0)

__device__ __forceinline__ v8i mk8(v4i a, v4i b) { return __builtin_shufflevector(a, b, 0, 1, 2, 3, 4, 5, 6, 7); }

__global__ void __launch_bounds__(512, 2) fused_gemm(Args a) {
  __shared__ __attribute__((aligned(1024))) unsigned char smem[LDS_BYTES];
  LAS unsigned char* lds = (LAS unsigned char*)smem;
  const int tid = threadIdx.x, lane = tid & 63, wid = __builtin_amdgcn_readfirstlane(tid >> 6);
  const int b = blockIdx.x;
  const int xl = b & 7, half = (b >> 3) & 1, tn = b >> 4;
  const int src = (a.rank + xl) & 7;
  const int tm = src * 2 + half;
  unsigned char* const myws = a.wsr[0];
  LAS unsigned* ctl = (LAS unsigned*)(lds + NSLOT * SLOT);

  const unsigned craw = __hip_atomic_load((unsigned*)(myws + WS_CTR), __ATOMIC_RELAXED, __HIP_MEMORY_SCOPE_AGENT);
  unsigned call;
  const bool exporter = (xl < 4);
  const int rw = (b >> 3) * 4 + (b & 3);
  unsigned char* ppeer = nullptr; unsigned* pflag = nullptr;
  if (exporter) {
    const int h = rw >> 6, r0 = (rw & 63) * 4;
    const __amdgpu_buffer_rsrc_t rs = __builtin_amdgcn_make_buffer_rsrc((void*)(a.x + (size_t)(h * 256 + r0) * K), 0, 4 * K, 0x00020000);
    u32x4 v0 = __builtin_amdgcn_raw_buffer_load_b128(rs, tid * 16, 0, 0);
    u32x4 v1 = __builtin_amdgcn_raw_buffer_load_b128(rs, tid * 16 + 8192, 0, 0);
    unsigned char* pl = a.wsr[lane & 7];
    call = __builtin_amdgcn_readfirstlane(craw) + 1u;
    const __amdgpu_buffer_rsrc_t rd = __builtin_amdgcn_make_buffer_rsrc((void*)(myws + WS_EXP + (call & 1u) * SHARD + (size_t)h * (SHARD / 2)), 0, (unsigned)(SHARD / 2), 0x00020000);
    const int o = tid * 16;
    const int row = r0 + (o >> 12), kt = (o & 4095) >> 7, c = o & 127;
    __builtin_amdgcn_raw_buffer_store_b128(v0, rd, kt * A_BYTES + row * BK + c, 0, 17);
    __builtin_amdgcn_raw_buffer_store_b128(v1, rd, kt * A_BYTES + (row + 2) * BK + c, 0, 17);
    asm volatile("s_waitcnt vmcnt(0)" ::: "memory");
    BAR();
    if (wid == 4 && lane < NR && lane != 0)
      __hip_atomic_store((unsigned*)(pl + WS_FLAGE) + a.rank * 128 + rw, call, __ATOMIC_RELAXED, __HIP_MEMORY_SCOPE_SYSTEM);
  } else if (wid < 4 && NPUSH > 0) {
    const int u = rw * 4 + wid;
    if (u < 64 * 7) {
      const int pc = u / 7, po = 1 + u % 7, ph = pc >> 5, pi = pc & 31;
      const __amdgpu_buffer_rsrc_t rsX = __builtin_amdgcn_make_buffer_rsrc((void*)(a.x + (size_t)(ph * 256 + pi * 8) * K), 0, 8 * K, 0x00020000);
      const unsigned vX = (lane >> 3) * K + (lane & 7) * 16;
      u32x4 v[NPUSH > 0 ? NPUSH : 1];
#pragma unroll
      for (int k = 0; k < NPUSH; ++k) v[k] = __builtin_amdgcn_raw_buffer_load_b128(rsX, vX, k * BK, 0);
      ppeer = a.wsr[po];
      call = __builtin_amdgcn_readfirstlane(craw) + 1u;
      const size_t so = WS_STG + (size_t)((call & 1u) * NR + a.rank) * STG_SRC + (size_t)(ph * 4) * A_BYTES + pi * 1024;
      const __amdgpu_buffer_rsrc_t rd = __builtin_amdgcn_make_buffer_rsrc((void*)(ppeer + so), 0, 4 * A_BYTES, 0x00020000);
#pragma unroll
      for (int k = 0; k < NPUSH; ++k) __builtin_amdgcn_raw_buffer_store_b128(v[k], rd, lane * 16, k * A_BYTES, 17);
      pflag = (unsigned*)(ppeer + WS_FLAGP) + a.rank * 64 + pc;
    }
  }
  call = __builtin_amdgcn_readfirstlane(craw) + 1u;
  const unsigned par = call & 1u;
  unsigned ticket = 0;

  if (wid < 4) {
    const int wm = wid >> 1, wn = wid & 1;
    const int r = lane & 15, g = lane >> 4;
    const unsigned aoff0 = (wm * 128 + r) * 128 + ((g ^ ((r >> 1) & 7)) * 16);
    const unsigned aoff1 = aoff0 ^ 64;
    const int q = r >> 1, p = r & 1;
    unsigned boff[4];
#pragma unroll
    for (int nn = 0; nn < 4; ++nn) boff[nn] = A_BYTES + 2 * g * 1024 + q * 128 + (((wn * 4 + nn) ^ ((q >> 1) | ((g & 1) << 2))) * 16) + p * 8;
    v4f acc[8][4];
#pragma unroll
    for (int m = 0; m < 8; ++m)
#pragma unroll
      for (int nn = 0; nn < 4; ++nn) acc[m][nn] = (v4f){0.f, 0.f, 0.f, 0.f};
    v8i Bf[2][4];
    v8i Af[4];
#define RDA(slotoff, m) mk8(*(const LAS v4i*)(lds + (slotoff) + aoff0 + (m) * 2048), *(const LAS v4i*)(lds + (slotoff) + aoff1 + (m) * 2048))
#define RDB2(slotoff, nn, r_) __builtin_amdgcn_ds_read_tr8_b64_v2i32((LAS v2i*)(lds + (slotoff) + boff[nn] + ((r_) & 1) * 1024 + ((r_) >> 1) * 8192))
#define RDB(slotoff, nn) ({ v2i b0 = RDB2(slotoff, nn, 0), b1 = RDB2(slotoff, nn, 1), b2 = RDB2(slotoff, nn, 2), b3 = RDB2(slotoff, nn, 3); \
      v4i lo = __builtin_shufflevector(b0, b1, 0, 1, 2, 3), hi = __builtin_shufflevector(b2, b3, 0, 1, 2, 3); mk8(lo, hi); })
#define MMA4(m, Bc, Ac) do { _Pragma("unroll") for (int nn = 0; nn < 4; ++nn) \
      acc[m][nn] = __builtin_amdgcn_mfma_scale_f32_16x16x128_f8f6f4(Bc[nn], Ac, acc[m][nn], 1, 0, 0, 0, 0, 0); } while (0)
    BAR();
    if (pflag) {
      asm volatile("s_waitcnt vmcnt(0)" ::: "memory");
      if (lane == 0) __hip_atomic_store(pflag, call, __ATOMIC_RELAXED, __HIP_MEMORY_SCOPE_SYSTEM);
    }
    BAR();
    {
      const unsigned so = 0;
#pragma unroll
      for (int nn = 0; nn < 4; ++nn) Bf[0][nn] = RDB(so, nn);
      Af[0] = RDA(so, 0); Af[1] = RDA(so, 1);
    }
    unsigned so = 0;
#define KSTEP(cur, nxt, LAST) do { \
      const unsigned sn = (so == 2 * SLOT) ? 0u : so + SLOT; \
      Af[2] = RDA(so, 2); SCHED(); MMA4(0, Bf[cur], Af[0]); SCHED(); \
      Af[3] = RDA(so, 3); SCHED(); MMA4(1, Bf[cur], Af[1]); SCHED(); \
      Af[0] = RDA(so, 4); SCHED(); MMA4(2, Bf[cur], Af[2]); SCHED(); \
      Af[1] = RDA(so, 5); SCHED(); MMA4(3, Bf[cur], Af[3]); SCHED(); \
      Af[2] = RDA(so, 6); Af[3] = RDA(so, 7); SCHED(); MMA4(4, Bf[cur], Af[0]); SCHED(); \
      MMA4(5, Bf[cur], Af[1]); SCHED(); \
      asm volatile("s_waitcnt lgkmcnt(0)" ::: "memory"); \
      BAR(); SCHED(); \
      if (!(LAST)) { Bf[nxt][0] = RDB(sn, 0); Bf[nxt][1] = RDB(sn, 1); Af[0] = RDA(sn, 0); } SCHED(); \
      MMA4(6, Bf[cur], Af[2]); SCHED(); \
      if (!(LAST)) { Bf[nxt][2] = RDB(sn, 2); Bf[nxt][3] = RDB(sn, 3); Af[1] = RDA(sn, 1); } SCHED(); \
      MMA4(7, Bf[cur], Af[3]); SCHED(); \
      so = sn; } while (0)
    for (int kt = 0; kt < NT - 2; kt += 2) {
      KSTEP(0, 1, false);
      KSTEP(1, 0, false);
    }
    KSTEP(0, 1, false);
    float s = a.sx[0] * a.sw[0];
    if (ctl[1]) s = __builtin_nanf("");
    const __amdgpu_buffer_rsrc_t rsC = __builtin_amdgcn_make_buffer_rsrc((void*)(a.out + (size_t)(tm * BM + wm * 128) * NS + tn * BN + wn * 64), 0, 128 * NS * 4, 0x00020000);
    const unsigned vC = (r & 3) * (NS * 4) + (r >> 2) * 64 + g * 16;
#define DPPF(old, src, ctrl, bank) __builtin_bit_cast(float, __builtin_amdgcn_update_dpp(__builtin_bit_cast(int, (float)(old)), __builtin_bit_cast(int, (float)(src)), ctrl, 0xf, bank, false))
#define OUTM(m) do { _Pragma("unroll") for (int e = 0; e < 4; ++e) { \
      const float r0 = acc[m][0][e] * s, r1 = acc[m][1][e] * s, r2 = acc[m][2][e] * s, r3 = acc[m][3][e] * s; \
      const float s0 = DPPF(r0, r2, 0x128, 0xc), s2 = DPPF(r2, r0, 0x128, 0x3); \
      const float s1 = DPPF(r1, r3, 0x128, 0xc), s3 = DPPF(r3, r1, 0x128, 0x3); \
      acc[m][0][e] = DPPF(s0, s1, 0x114, 0xa); acc[m][1][e] = DPPF(s1, s0, 0x104, 0x5); \
      acc[m][2][e] = DPPF(s2, s3, 0x114, 0xa); acc[m][3][e] = DPPF(s3, s2, 0x104, 0x5); } \
      _Pragma("unroll") for (int t = 0; t < 4; ++t) \
        __builtin_amdgcn_raw_buffer_store_b128(__builtin_bit_cast(u32x4, acc[m][t]), rsC, vC, ((m) * 16 + t * 4) * (NS * 4), 16); \
      SCHED(); } while (0)
    Af[2] = RDA(so, 2); SCHED(); MMA4(0, Bf[1], Af[0]); SCHED();
    Af[3] = RDA(so, 3); SCHED(); MMA4(1, Bf[1], Af[1]); SCHED();
    Af[0] = RDA(so, 4); SCHED(); MMA4(2, Bf[1], Af[2]); SCHED(); OUTM(0);
    Af[1] = RDA(so, 5); SCHED(); MMA4(3, Bf[1], Af[3]); SCHED(); OUTM(1);
    Af[2] = RDA(so, 6); Af[3] = RDA(so, 7); SCHED(); MMA4(4, Bf[1], Af[0]); SCHED(); OUTM(2);
    MMA4(5, Bf[1], Af[1]); SCHED(); OUTM(3);
    MMA4(6, Bf[1], Af[2]); SCHED(); OUTM(4);
    MMA4(7, Bf[1], Af[3]); SCHED(); OUTM(5);
    OUTM(6);
    OUTM(7);
  } else {
    const int hw = wid - 4;
    const bool local = (src == a.rank);
    const int pos0 = PBASE + (tn & 7);
    if (hw == 0) {
      unsigned* const tmo = (unsigned*)(myws + WS_TMO);
      const unsigned long long t0 = __builtin_amdgcn_s_memrealtime();
#define WAIT_FLAGS(f) do { for (;;) { \
          const unsigned v_ = __hip_atomic_load(f, __ATOMIC_RELAXED, __HIP_MEMORY_SCOPE_SYSTEM); \
          const unsigned tv_ = __hip_atomic_load(tmo, __ATOMIC_RELAXED, __HIP_MEMORY_SCOPE_AGENT); \
          if (__all(v_ >= call)) break; \
          if (tv_ == call || __builtin_amdgcn_s_memrealtime() - t0 > DEADLINE_TICKS) { \
            if (lane == 0) __hip_atomic_store(tmo, call, __ATOMIC_RELAXED, __HIP_MEMORY_SCOPE_AGENT); \
            asm volatile("s_waitcnt vmcnt(0)" ::: "memory"); \
            break; \
          } } } while (0)
#define WAIT_FLAGS2(f) do { for (;;) { \
          const unsigned v_ = __hip_atomic_load(f, __ATOMIC_RELAXED, __HIP_MEMORY_SCOPE_SYSTEM); \
          const unsigned w_ = __hip_atomic_load(f + 64, __ATOMIC_RELAXED, __HIP_MEMORY_SCOPE_SYSTEM); \
          const unsigned tv_ = __hip_atomic_load(tmo, __ATOMIC_RELAXED, __HIP_MEMORY_SCOPE_AGENT); \
          if (__all(v_ >= call && w_ >= call)) break; \
          if (tv_ == call || __builtin_amdgcn_s_memrealtime() - t0 > DEADLINE_TICKS) { \
            if (lane == 0) __hip_atomic_store(tmo, call, __ATOMIC_RELAXED, __HIP_MEMORY_SCOPE_AGENT); \
            asm volatile("s_waitcnt vmcnt(0)" ::: "memory"); \
            break; \
          } } } while (0)
      const unsigned* fE = (const unsigned*)(myws + WS_FLAGE) + src * 128 + lane;
      const unsigned* fP = (const unsigned*)(myws + WS_FLAGP) + src * 64 + half * 32 + (lane & 31);
      if (!local) {
        WAIT_FLAGS2(fE);
      }
      BAR();
      if (lane == 0) ticket = __hip_atomic_fetch_add((unsigned*)(myws + WS_TICK), 1u, __ATOMIC_RELAXED, __HIP_MEMORY_SCOPE_AGENT);
      BAR();
      const int JW = pos0 - 3;
      int js = 0;
      if (!local && NPUSH > 0) {
        bool seen = false;
        while (js + 2 <= JW) {
          const unsigned vP = __hip_atomic_load(fP, __ATOMIC_RELAXED, __HIP_MEMORY_SCOPE_SYSTEM);
          BAR();
          seen = __all(vP >= call);
          BAR();
          js += 2;
          if (seen) break;
        }
        if (!seen) {
          for (; js < JW; ++js) BAR();
          WAIT_FLAGS(fP);
        }
      }
      {
        const unsigned tv2 = __hip_atomic_load(tmo, __ATOMIC_RELAXED, __HIP_MEMORY_SCOPE_AGENT);
        BAR(); ++js;
        if (lane == 0) ctl[1] = (tv2 == call);
        asm volatile("s_waitcnt lgkmcnt(0)" ::: "memory");
      }
      for (; js < NT - 1; ++js) BAR();
    } else {
      const unsigned char* abase = local ? (a.x + (size_t)half * BM * K) : (a.wsr[xl] + WS_EXP + par * SHARD + (size_t)half * (SHARD / 2));
      const unsigned rowstride = local ? K : BK;
      const unsigned kstride = local ? BK : A_BYTES;
      const __amdgpu_buffer_rsrc_t rsA = __builtin_amdgcn_make_buffer_rsrc((void*)abase, 0, BM * K, 0x00020000);
      const __amdgpu_buffer_rsrc_t rsS = __builtin_amdgcn_make_buffer_rsrc((void*)(myws + WS_STG + (size_t)(par * NR + src) * STG_SRC + (size_t)half * 4 * A_BYTES), 0, 4 * A_BYTES, 0x00020000);
      const __amdgpu_buffer_rsrc_t rsB = __builtin_amdgcn_make_buffer_rsrc((void*)(a.W + a.rank * NS + tn * BN), 0, (unsigned)((size_t)K * NW - (a.rank * NS + tn * BN)), 0x00020000);
      const __amdgpu_buffer_rsrc_t rsO = __builtin_amdgcn_make_buffer_rsrc((void*)(a.wsr[xl] + WS_EXP + par * SHARD + (size_t)(half ^ 1) * (SHARD / 2)), 0, BM * K, 0x00020000);
      const int lq = lane >> 3, lc = lane & 7;
      const unsigned vA0 = lq * rowstride + ((lc ^ (lane >> 4)) * 16);
      const unsigned vA1 = lq * rowstride + ((lc ^ (4 + (lane >> 4))) * 16);
      const unsigned vB0 = lq * NW + ((lc ^ (lq >> 1)) * 16);
      const unsigned vB1 = lq * NW + ((lc ^ ((lq >> 1) | 4)) * 16);
      const unsigned vO = lane * 16;
      int nj = 0, nkt = local ? 0 : NPUSH + (xl * ROT) % (NT - NPUSH), npi = 0, npp = (local || NPUSH == 0) ? 1000 : pos0;
      unsigned islot = 0;
#define DMA_A(rs, slotoff, i, so) __builtin_amdgcn_raw_ptr_buffer_load_lds(rs, (LAS void*)(lds + (slotoff) + (i) * 1024), 16, ((i) & 1) ? vA1 : vA0, (so) + (i) * 8 * rowstride, 0, 0)
#define DMA_B(slotoff, j, kt) __builtin_amdgcn_raw_ptr_buffer_load_lds(rsB, (LAS void*)(lds + (slotoff) + A_BYTES + (j) * 1024), 16, (((j) >> 1) & 1) ? vB1 : vB0, ((kt) * BK + (j) * 8) * NW, 0, 0)
#define ISSUE() do { \
        int kt_; bool ps_ = false; \
        if (nj == npp) { kt_ = npi; ps_ = true; ++npi; npp = (npi < NPUSH) ? npp + PSTR : 1000; } else { kt_ = nkt; ++nkt; if (nkt == NT && !local) nkt = NPUSH; } \
        if (hw == 3) { _Pragma("unroll") for (int j = 0; j < 16; ++j) DMA_B(islot, j, kt_); } \
        else { \
          const unsigned lo_ = islot + (hw - 1) * 16384, so_ = kt_ * kstride + (hw - 1) * 128 * rowstride; \
          if (ps_) { _Pragma("unroll") for (int i = 0; i < 16; ++i) DMA_A(rsS, lo_, i, so_); } \
          else { _Pragma("unroll") for (int i = 0; i < 16; ++i) DMA_A(rsA, lo_, i, so_); } \
        } \
        ++nj; islot += SLOT; if (islot == NSLOT * SLOT) islot = 0; } while (0)
#define PF_O(kt) __builtin_amdgcn_raw_ptr_buffer_load_lds(rsO, (LAS void*)(lds + NSLOT * SLOT + 2048 + (hw - 1) * 1024), 16, vO, (kt) * A_BYTES + (tn * 2 + hw - 1) * 1024, 0, 0)
      if (hw == 3) { ISSUE(); ISSUE(); ISSUE(); }
      BAR();
      if (hw != 3) {
        if (!local && half == 1) {
#pragma unroll
          for (int d = 0; d < DE; ++d) PF_O(NPUSH + 3 + d);
          if (HOLD1) asm volatile("s_waitcnt vmcnt(0)" ::: "memory");
        }
        ISSUE(); ISSUE(); ISSUE();
      }
      asm volatile("s_waitcnt vmcnt(32)" ::: "memory");
      BAR();
      for (int js = 0; js < NT - 3; ++js) {
        asm volatile("s_waitcnt vmcnt(16)" ::: "memory");
        BAR();
        ISSUE();
      }
      if (DL > 0 && !local && half == 0 && hw != 3) {
#pragma unroll
        for (int d = 0; d < DL; ++d) PF_O(NT - DL + d);
        asm volatile("s_waitcnt vmcnt(%0)" :: "n"(16 + DL) : "memory");
        BAR();
        asm volatile("s_waitcnt vmcnt(%0)" :: "n"(DL) : "memory");
        BAR();
      } else {
        asm volatile("s_waitcnt vmcnt(16)" ::: "memory");
        BAR();
        asm volatile("s_waitcnt vmcnt(0)" ::: "memory");
        BAR();
      }
    }
  }
  if (tid == 256 && (ticket % NWG) == NWG - 1)
    __hip_atomic_store((unsigned*)(myws + WS_CTR), call, __ATOMIC_RELAXED, __HIP_MEMORY_SCOPE_AGENT);
}

}

extern "C" void kernel_launch(int rank, int nranks, ncclComm_t comm,
                              hipStream_t stream,
                              void* const* d_in, const int* in_sizes, int n_in,
                              void* d_out, int out_size,
                              const int* in_dtypes, int out_dtype,
                              void* const* d_out_all,
                              void* const* d_ws_all, size_t ws_bytes) {
  if (nranks != NR || n_in < 4) { fprintf(stderr, "unsupported configuration\n"); return; }
  Args a;
  a.x = (const unsigned char*)d_in[0];
  a.W = (const unsigned char*)d_in[1];
  a.sx = (const float*)d_in[2];
  a.sw = (const float*)d_in[3];
  a.out = (float*)d_out;
  for (int p = 0; p < NR; ++p) a.wsr[p] = (unsigned char*)d_ws_all[(rank + p) & 7];
  a.rank = rank;
  hipLaunchKernelGGL(fused_gemm, dim3(NWG), dim3(512), 0, stream, a);
  HIP_CHECK(hipGetLastError());
}
```

```cpp
#include <hip/hip_runtime.h>
#include <rccl/rccl.h>
#include <cstdio>

#define HIP_CHECK(expr) do { hipError_t _e = (expr); if (_e != hipSuccess) { \
    fprintf(stderr, "HIP error %s at %s:%d\n", hipGetErrorString(_e), __FILE__, __LINE__); \
    return; } } while (0)

namespace {

typedef int v8i __attribute__((ext_vector_type(8)));
typedef int v4i __attribute__((ext_vector_type(4)));
typedef int v2i __attribute__((ext_vector_type(2)));
typedef float v4f __attribute__((ext_vector_type(4)));
typedef unsigned u32x4 __attribute__((ext_vector_type(4)));
#define LAS __attribute__((address_space(3)))

constexpr int NR = 8;
constexpr int M = 4096, K = 4096, NW = 16384, NS = 2048, MS = 512;
constexpr int BM = 256, BN = 128, BK = 128, NT = K / BK;
constexpr int A_BYTES = BM * BK, B_BYTES = BK * BN, SLOT = A_BYTES + B_BYTES;
constexpr int NSLOT = 3;
constexpr int LDS_BYTES = NSLOT * SLOT + B_BYTES;
constexpr int NWG = 256;
#define NPUSHV 2
#define PBASE 6
constexpr int NPUSH = NPUSHV;
constexpr int PSTR = NPUSH <= 3 ? 8 : 6;
constexpr int NE = NT - 4;
static_assert(NE % NSLOT == 1, "the region tables below assume that step NE-3 sits in slot 1");
__host__ __device__ constexpr unsigned g_off(int g) {
  return g < 2 ? 49152u + g * 8192u : g < 6 ? 98304u + (g - 2) * 8192u : g < 12 ? (g - 6) * 8192u : g < 14 ? 114688u + (g - 12) * 8192u : 49152u + (g - 14) * 8192u; }
__host__ __device__ constexpr unsigned gb_off(int j) { return j == 0 ? 147456u : j == 1 ? 65536u : j == 2 ? 81920u : 131072u; }
#define GD 4
__host__ __device__ constexpr int g_avail(int g) { return g < 2 ? -3 : g < 6 ? -2 : g < 12 ? -1 : g == 12 ? 4 : 5; }
#define GD2 GD
#define GSW 8
__host__ __device__ constexpr int g_issue(int g) {
  int m = -3;
  for (int q = 0; q <= g; ++q) { const int d = q - 1 - (q < GSW ? GD : GD2); const int v = g_avail(q) > d ? g_avail(q) : d; if (v > m) m = v; }
  return m; }
__host__ __device__ constexpr int g_allow(int sidx) {
  int n = 0;
  for (int g = 0; g < 16; ++g) if (g_issue(g) < sidx && g > sidx + 1) ++n;
  return 4 * n; }
__host__ __device__ constexpr unsigned st_off(int p, int w) {
  return p == 0 ? (w ? 98304u : 49152u) : p == 2 ? (w ? 32768u : 16384u) : (w ? 0u : 98304u); }

constexpr size_t WS_CTR = 0, WS_TICK = 128, WS_TMO = 256;
constexpr size_t WS_FLAGE = 65536;
constexpr size_t WS_FLAGP = 65536 + 16384;
constexpr size_t WS_STAMP = 131072;
constexpr size_t WS_EXP = 1u << 20;
constexpr size_t WS_STG = 8u << 20;
constexpr size_t SHARD = (size_t)MS * K;
constexpr unsigned STG_SRC = 2 * 4 * A_BYTES;
constexpr unsigned long long DEADLINE_TICKS = 50000000ull;

struct Args {
  const unsigned char* x; const unsigned char* W; const float* sx; const float* sw; float* out;
  unsigned char* wsr[NR];
  int rank;
};

#define BAR() __builtin_amdgcn_s_barrier()
#define DBGNOP() do {} while (0)
#define SCHED() __builtin_amdgcn_sched_barrier(0)
#define WAIT_LGKM0() do { asm volatile("" ::: "memory"); __builtin_amdgcn_s_waitcnt(0xC07F); asm volatile("" ::: "memory"); } while (0)
#define BARM() do { asm volatile("" ::: "memory"); __builtin_amdgcn_s_barrier(); asm volatile("" ::: "memory"); } while (0)
#define WAIT_VM(n) asm volatile("s_waitcnt vmcnt(%0)" :: "n"(n) : "memory")

__device__ __forceinline__ v8i mk8(v4i a, v4i b) { return __builtin_shufflevector(a, b, 0, 1, 2, 3, 4, 5, 6, 7); }

__global__ void __launch_bounds__(512, 2) fused_gemm(Args a) {
  __shared__ __attribute__((aligned(1024))) unsigned char smem[LDS_BYTES];
  LAS unsigned char* lds = (LAS unsigned char*)smem;
  const int tid = threadIdx.x, lane = tid & 63, wid = __builtin_amdgcn_readfirstlane(tid >> 6);
  const int b = blockIdx.x;
  const int xl = b & 7, half = (b >> 3) & 1, tn = b >> 4;
  const int src = (a.rank + xl) & 7;
  const int tm = src * 2 + half;
  unsigned char* const myws = a.wsr[0];

  const unsigned craw = __hip_atomic_load((unsigned*)(myws + WS_CTR), __ATOMIC_RELAXED, __HIP_MEMORY_SCOPE_AGENT);
  unsigned call;
  const bool exporter = (xl < 4);
  const int rw = (b >> 3) * 4 + (b & 3);
  unsigned char* ppeer = nullptr; unsigned* pflag = nullptr;
  if (exporter) {
    const int h = rw >> 6, r0 = (rw & 63) * 4;
    const __amdgpu_buffer_rsrc_t rs = __builtin_amdgcn_make_buffer_rsrc((void*)(a.x + (size_t)(h * 256 + r0) * K), 0, 4 * K, 0x00020000);
    u32x4 v0 = __builtin_amdgcn_raw_buffer_load_b128(rs, tid * 16, 0, 0);
    u32x4 v1 = __builtin_amdgcn_raw_buffer_load_b128(rs, tid * 16 + 8192, 0, 0);
    unsigned char* pl = a.wsr[lane & 7];
    call = __builtin_amdgcn_readfirstlane(craw) + 1u;
    const __amdgpu_buffer_rsrc_t rd = __builtin_amdgcn_make_buffer_rsrc((void*)(myws + WS_EXP + (call & 1u) * SHARD + (size_t)h * (SHARD / 2)), 0, (unsigned)(SHARD / 2), 0x00020000);
    const int o = tid * 16;
    const int row = r0 + (o >> 12), kt = (o & 4095) >> 7, c = o & 127;
#define EXP_OFF(row_) (kt < NE ? kt * A_BYTES + (row_) * BK : NE * A_BYTES + (((((row_) >> 5) & 3) * 4 + (kt - NE)) * 2 + ((row_) >> 7)) * 4096 + ((row_) & 31) * BK)
    __builtin_amdgcn_raw_buffer_store_b128(v0, rd, EXP_OFF(row) + c, 0, 17);
    __builtin_amdgcn_raw_buffer_store_b128(v1, rd, EXP_OFF(row + 2) + c, 0, 17);
    asm volatile("s_waitcnt vmcnt(0)" ::: "memory");
    BAR();
    if (wid == 4 && lane < NR && lane != 0)
      __hip_atomic_store((unsigned*)(pl + WS_FLAGE) + a.rank * 128 + rw, call, __ATOMIC_RELAXED, __HIP_MEMORY_SCOPE_SYSTEM);
  } else if (wid < 4 && NPUSH > 0) {
    const int u = rw * 4 + wid;
    if (u < 64 * 7) {
      const int pc = u / 7, po = 1 + u % 7, ph = pc >> 5, pi = pc & 31;
      const __amdgpu_buffer_rsrc_t rsX = __builtin_amdgcn_make_buffer_rsrc((void*)(a.x + (size_t)(ph * 256 + pi * 8) * K), 0, 8 * K, 0x00020000);
      const unsigned vX = (lane >> 3) * K + (lane & 7) * 16;
      u32x4 v[NPUSH > 0 ? NPUSH : 1];
#pragma unroll
      for (int k = 0; k < NPUSH; ++k) v[k] = __builtin_amdgcn_raw_buffer_load_b128(rsX, vX, k * BK, 0);
      ppeer = a.wsr[po];
      call = __builtin_amdgcn_readfirstlane(craw) + 1u;
      const size_t so = WS_STG + (size_t)((call & 1u) * NR + a.rank) * STG_SRC + (size_t)(ph * 4) * A_BYTES + pi * 1024;
      const __amdgpu_buffer_rsrc_t rd = __builtin_amdgcn_make_buffer_rsrc((void*)(ppeer + so), 0, 4 * A_BYTES, 0x00020000);
#pragma unroll
      for (int k = 0; k < NPUSH; ++k) __builtin_amdgcn_raw_buffer_store_b128(v[k], rd, lane * 16, k * A_BYTES, 17);
      pflag = (unsigned*)(ppeer + WS_FLAGP) + a.rank * 64 + pc;
    }
  }
  call = __builtin_amdgcn_readfirstlane(craw) + 1u;
  const unsigned par = call & 1u;
  unsigned ticket = 0;

  if (wid < 4) {
    const int wm = wid >> 1, wn = wid & 1;
    const int r = lane & 15, g = lane >> 4;
    const unsigned aoff0 = (wm * 128 + r) * 128 + ((g ^ ((r >> 1) & 7)) * 16);
    const unsigned aoff1 = aoff0 ^ 64;
    const int q = r >> 1, p = r & 1;
    unsigned boff[4];
#pragma unroll
    for (int nn = 0; nn < 4; ++nn) boff[nn] = A_BYTES + 2 * g * 1024 + q * 128 + (((wn * 4 + nn) ^ ((q >> 1) | ((g & 1) << 2))) * 16) + p * 8;
    v4f acc[8][4];
#pragma unroll
    for (int m = 0; m < 8; ++m)
#pragma unroll
      for (int nn = 0; nn < 4; ++nn) acc[m][nn] = (v4f){0.f, 0.f, 0.f, 0.f};
    v8i Bf[2][4];
    v8i Af[4];
#define RDA(slotoff, m) mk8(*(const LAS v4i*)(lds + (slotoff) + aoff0 + (m) * 2048), *(const LAS v4i*)(lds + (slotoff) + aoff1 + (m) * 2048))
#define RDB2(slotoff, nn, r_) __builtin_amdgcn_ds_read_tr8_b64_v2i32((LAS v2i*)(lds + (slotoff) + boff[nn] + ((r_) & 1) * 1024 + ((r_) >> 1) * 8192))
#define RDB(slotoff, nn) ({ v2i b0 = RDB2(slotoff, nn, 0), b1 = RDB2(slotoff, nn, 1), b2 = RDB2(slotoff, nn, 2), b3 = RDB2(slotoff, nn, 3); \
      v4i lo = __builtin_shufflevector(b0, b1, 0, 1, 2, 3), hi = __builtin_shufflevector(b2, b3, 0, 1, 2, 3); mk8(lo, hi); })
#define MMA4(m, Bc, Ac) do { _Pragma("unroll") for (int nn = 0; nn < 4; ++nn) \
      acc[m][nn] = __builtin_amdgcn_mfma_scale_f32_16x16x128_f8f6f4(Bc[nn], Ac, acc[m][nn], 1, 0, 0, 0, 0, 0); } while (0)
    BAR();
    if (pflag) {
      asm volatile("s_waitcnt vmcnt(0)" ::: "memory");
      if (lane == 0) __hip_atomic_store(pflag, call, __ATOMIC_RELAXED, __HIP_MEMORY_SCOPE_SYSTEM);
    }
    BAR();
    {
      const unsigned so = 0;
#pragma unroll
      for (int nn = 0; nn < 4; ++nn) Bf[0][nn] = RDB(so, nn);
      Af[0] = RDA(so, 0); Af[1] = RDA(so, 1);
    }
    unsigned so = 0;
#define KSTEP(cur, nxt, ADJ, BADJ) do { \
      const unsigned sn = (so == 2 * SLOT) ? 0u : so + SLOT; \
      const unsigned sa = sn - (ADJ), sb = sn + (BADJ); \
      Af[2] = RDA(so, 2); SCHED(); MMA4(0, Bf[cur], Af[0]); SCHED(); \
      Af[3] = RDA(so, 3); SCHED(); MMA4(1, Bf[cur], Af[1]); SCHED(); \
      Af[0] = RDA(so, 4); SCHED(); MMA4(2, Bf[cur], Af[2]); SCHED(); \
      Af[1] = RDA(so, 5); SCHED(); MMA4(3, Bf[cur], Af[3]); SCHED(); \
      Af[2] = RDA(so, 6); Af[3] = RDA(so, 7); SCHED(); MMA4(4, Bf[cur], Af[0]); SCHED(); \
      MMA4(5, Bf[cur], Af[1]); SCHED(); \
      asm volatile("s_waitcnt lgkmcnt(0)" ::: "memory"); \
      BAR(); SCHED(); \
      Bf[nxt][0] = RDB(sb, 0); Bf[nxt][1] = RDB(sb, 1); Af[0] = RDA(sa, 0); SCHED(); \
      MMA4(6, Bf[cur], Af[2]); SCHED(); \
      Bf[nxt][2] = RDB(sb, 2); Bf[nxt][3] = RDB(sb, 3); Af[1] = RDA(sa, 1); SCHED(); \
      MMA4(7, Bf[cur], Af[3]); SCHED(); \
      so = sn; } while (0)
    static_assert(g_off(0) == (NE % NSLOT) * SLOT, "granule 0 must sit in the ring slot of step NE");
    for (int kt = 0; kt < NE; kt += 2) {
      KSTEP(0, 1, 0u, 0u);
      KSTEP(1, 0, (kt == NE - 2) ? wm * 12288u : 0u, (kt == NE - 2) ? gb_off(0) - (g_off(0) + A_BYTES) : 0u);
    }
    unsigned bq[4], ga0 = aoff0 - wm * 12288u + g_off(1), ga1 = ga0 ^ 64u;
#pragma unroll
    for (int nn = 0; nn < 4; ++nn) { bq[nn] = boff[nn] + (gb_off(1) - A_BYTES); asm volatile("" : "+v"(bq[nn])); }
    asm volatile("" : "+v"(ga0), "+v"(ga1));
    const unsigned d0 = r * 512 + wn * 256 + ((g ^ r) * 16);
#define GRDA(mm) mk8(*(const LAS v4i*)(lds + ga0 + (mm) * 2048), *(const LAS v4i*)(lds + ga1 + (mm) * 2048))
#define GRDB2(nn, r_) __builtin_amdgcn_ds_read_tr8_b64_v2i32((LAS v2i*)(lds + bq[nn] + ((r_) & 1) * 1024 + ((r_) >> 1) * 8192))
#define GRDB(nn) ({ v2i b0 = GRDB2(nn, 0), b1 = GRDB2(nn, 1), b2 = GRDB2(nn, 2), b3 = GRDB2(nn, 3); \
      v4i lo = __builtin_shufflevector(b0, b1, 0, 1, 2, 3), hi = __builtin_shufflevector(b2, b3, 0, 1, 2, 3); mk8(lo, hi); })
#define MM1(m, nn, Bc, Ac) do { acc[m][nn] = __builtin_amdgcn_mfma_scale_f32_16x16x128_f8f6f4(Bc[nn], Ac, acc[m][nn], 1, 0, 0, 0, 0, 0); SCHED(); } while (0)
#define PSTEP(s) do { \
      WAIT_LGKM0(); BARM(); SCHED(); \
      MM1(((s) >> 2) * 2, 0, Bf[(s) & 1], Af[((s) & 1) * 2]); \
      if ((s) < 15) { Bf[((s) + 1) & 1][0] = GRDB(0); SCHED(); } \
      MM1(((s) >> 2) * 2, 1, Bf[(s) & 1], Af[((s) & 1) * 2]); \
      if ((s) < 15) { Af[(((s) + 1) & 1) * 2] = GRDA(0); SCHED(); } \
      MM1(((s) >> 2) * 2, 2, Bf[(s) & 1], Af[((s) & 1) * 2]); \
      if ((s) < 15) { Bf[((s) + 1) & 1][1] = GRDB(1); SCHED(); } \
      MM1(((s) >> 2) * 2, 3, Bf[(s) & 1], Af[((s) & 1) * 2]); \
      if ((s) < 15) { Bf[((s) + 1) & 1][2] = GRDB(2); SCHED(); } \
      MM1(((s) >> 2) * 2 + 1, 0, Bf[(s) & 1], Af[((s) & 1) * 2 + 1]); \
      if ((s) < 15) { Bf[((s) + 1) & 1][3] = GRDB(3); SCHED(); } \
      MM1(((s) >> 2) * 2 + 1, 1, Bf[(s) & 1], Af[((s) & 1) * 2 + 1]); \
      if ((s) < 15) { Af[(((s) + 1) & 1) * 2 + 1] = GRDA(1); SCHED(); } \
      MM1(((s) >> 2) * 2 + 1, 2, Bf[(s) & 1], Af[((s) & 1) * 2 + 1]); \
      if ((s) < 15) { _Pragma("unroll") for (int nn = 0; nn < 4; ++nn) { bq[nn] += gb_off(((s) + 2) & 3) - gb_off(((s) + 1) & 3); asm volatile("" : "+v"(bq[nn])); } \
        ga0 += g_off((s) < 14 ? (s) + 2 : 15) - g_off((s) + 1); ga1 += g_off((s) < 14 ? (s) + 2 : 15) - g_off((s) + 1); asm volatile("" : "+v"(ga0), "+v"(ga1)); SCHED(); } \
      MM1(((s) >> 2) * 2 + 1, 3, Bf[(s) & 1], Af[((s) & 1) * 2 + 1]); \
      if (((s) & 3) == 3) { unsigned db = d0 + (wm ? st_off((s) >> 2, 1) : st_off((s) >> 2, 0)); asm volatile("" : "+v"(db)); \
        _Pragma("unroll") for (int nn = 0; nn < 4; ++nn) _Pragma("unroll") for (int mm = 0; mm < 2; ++mm) \
          *(LAS v4f*)(lds + (db ^ (nn * 64)) + mm * 8192) = acc[((s) >> 2) * 2 + mm][nn]; SCHED(); } \
      } while (0)
    PSTEP(0); PSTEP(1); PSTEP(2); PSTEP(3);
    PSTEP(4); PSTEP(5); PSTEP(6); PSTEP(7);
    PSTEP(8); PSTEP(9); PSTEP(10); PSTEP(11);
    PSTEP(12); PSTEP(13); PSTEP(14);
    PSTEP(15);
    WAIT_LGKM0(); BARM();
  } else {
    const int hw = wid - 4;
    const bool local = (src == a.rank);
    const int pos0 = PBASE + (tn & 7);
    unsigned* const tmo = (unsigned*)(myws + WS_TMO);
    const __amdgpu_buffer_rsrc_t rsC = __builtin_amdgcn_make_buffer_rsrc((void*)(a.out + (size_t)(tm * BM) * NS + tn * BN), 0, BM * NS * 4, 0x00020000);
    const unsigned vG = (lane >> 5) * (NS * 4) + (lane & 31) * 16;
    const unsigned vL = (lane >> 5) * 512 + (((lane & 31) ^ (lane >> 5)) * 16);
    const float sc = a.sx[0] * a.sw[0];
    float s;
    v4f d[16];
    const __amdgpu_buffer_rsrc_t rsB = __builtin_amdgcn_make_buffer_rsrc((void*)(a.W + a.rank * NS + tn * BN), 0, (unsigned)((size_t)K * NW - (a.rank * NS + tn * BN)), 0x00020000);
    const int lq = lane >> 3, lc = lane & 7;
    const unsigned vB0 = lq * NW + ((lc ^ (lq >> 1)) * 16);
    const unsigned vB1 = lq * NW + ((lc ^ ((lq >> 1) | 4)) * 16);
#define DMA_B(boffs, j, kt) __builtin_amdgcn_raw_ptr_buffer_load_lds(rsB, (LAS void*)(lds + (boffs) + (j) * 1024), 16, (((j) >> 1) & 1) ? vB1 : vB0, ((kt) * BK + (j) * 8) * NW, 0, 0)
#define ISSUE_B(j, j0, j1) do { _Pragma("unroll") for (int jj = (j0); jj < (j1); ++jj) DMA_B(gb_off(j), jj, NE + (j)); } while (0)
#define ORD(pb, sw, i) (*(const LAS v4f*)(lds + st_off(pb, sw) + (i) * 1024 + (vL ^ (((2 * (i)) & 15) * 16))))
#define OST(pb, sw, i, v) __builtin_amdgcn_raw_buffer_store_b128(__builtin_bit_cast(u32x4, v), rsC, vG, ((sw) * 128 + (pb) * 32 + 2 * (i)) * (NS * 4), 16)
#define STNOP() do { SCHED(); asm volatile("s_nop 1"); SCHED(); } while (0)
#define DRAIN(pb, sw, LASTB) do { \
      _Pragma("unroll") for (int i = 0; i < 16; ++i) d[i] = ORD(pb, sw, i); \
      _Pragma("unroll") for (int i = 0; i < 16; ++i) { d[i] = d[i] * s; asm volatile("" : "+v"(d[i])); } \
      SCHED(); \
      _Pragma("unroll") for (int i = 0; i < 4; ++i) OST(pb, sw, i, d[i]); \
      STNOP(); WAIT_LGKM0(); BARM(); \
      _Pragma("unroll") for (int i = 4; i < 8; ++i) OST(pb, sw, i, d[i]); \
      STNOP(); BARM(); \
      _Pragma("unroll") for (int i = 8; i < 12; ++i) OST(pb, sw, i, d[i]); \
      STNOP(); BARM(); \
      _Pragma("unroll") for (int i = 12; i < 16; ++i) OST(pb, sw, i, d[i]); \
      STNOP(); \
      if (LASTB) BARM(); } while (0)
    if (hw == 0) {
      const unsigned long long t0 = __builtin_amdgcn_s_memrealtime();
#define WAIT_FLAGS(f) do { for (;;) { \
          const unsigned v_ = __hip_atomic_load(f, __ATOMIC_RELAXED, __HIP_MEMORY_SCOPE_SYSTEM); \
          const unsigned tv_ = __hip_atomic_load(tmo, __ATOMIC_RELAXED, __HIP_MEMORY_SCOPE_AGENT); \
          if (__all(v_ >= call)) break; \
          if (tv_ == call || __builtin_amdgcn_s_memrealtime() - t0 > DEADLINE_TICKS) { \
            if (lane == 0) __hip_atomic_store(tmo, call, __ATOMIC_RELAXED, __HIP_MEMORY_SCOPE_AGENT); \
            asm volatile("s_waitcnt vmcnt(0)" ::: "memory"); \
            break; \
          } } } while (0)
#define WAIT_FLAGS2(f) do { for (;;) { \
          const unsigned v_ = __hip_atomic_load(f, __ATOMIC_RELAXED, __HIP_MEMORY_SCOPE_SYSTEM); \
          const unsigned w_ = __hip_atomic_load(f + 64, __ATOMIC_RELAXED, __HIP_MEMORY_SCOPE_SYSTEM); \
          const unsigned tv_ = __hip_atomic_load(tmo, __ATOMIC_RELAXED, __HIP_MEMORY_SCOPE_AGENT); \
          if (__all(v_ >= call && w_ >= call)) break; \
          if (tv_ == call || __builtin_amdgcn_s_memrealtime() - t0 > DEADLINE_TICKS) { \
            if (lane == 0) __hip_atomic_store(tmo, call, __ATOMIC_RELAXED, __HIP_MEMORY_SCOPE_AGENT); \
            asm volatile("s_waitcnt vmcnt(0)" ::: "memory"); \
            break; \
          } } } while (0)
      const unsigned* fE = (const unsigned*)(myws + WS_FLAGE) + src * 128 + lane;
      const unsigned* fP = (const unsigned*)(myws + WS_FLAGP) + src * 64 + half * 32 + (lane & 31);
      if (!local) {
        WAIT_FLAGS2(fE);
      }
      BAR();
      if (lane == 0) ticket = __hip_atomic_fetch_add((unsigned*)(myws + WS_TICK), 1u, __ATOMIC_RELAXED, __HIP_MEMORY_SCOPE_AGENT);
      BAR();
      const int JW = pos0 - 3;
      int js = 0;
      if (!local && NPUSH > 0) {
        bool seen = false;
        while (js + 2 <= JW) {
          const unsigned vP = __hip_atomic_load(fP, __ATOMIC_RELAXED, __HIP_MEMORY_SCOPE_SYSTEM);
          BAR();
          seen = __all(vP >= call);
          BAR();
          js += 2;
          if (seen) break;
        }
        if (!seen) {
          for (; js < JW; ++js) BAR();
          WAIT_FLAGS(fP);
        }
      }
      for (; js < NE - 3; ++js) BAR();
      ISSUE_B(0, 0, 16);
      BAR(); BAR();
      ISSUE_B(3, 0, 8);
      unsigned tv2 = __hip_atomic_load(tmo, __ATOMIC_RELAXED, __HIP_MEMORY_SCOPE_AGENT);
      WAIT_VM(9); BAR();
      BAR(); BAR();
      WAIT_VM(0); BAR();
      asm volatile("" : "+v"(tv2));
      s = (tv2 == call) ? __builtin_nanf("") : sc;
      BAR(); BARM();
      DRAIN(0, 0, true); DRAIN(1, 0, true); DRAIN(2, 0, false);
    } else {
      const unsigned char* abase = local ? (a.x + (size_t)half * BM * K) : (a.wsr[xl] + WS_EXP + par * SHARD + (size_t)half * (SHARD / 2));
      const unsigned rowstride = local ? K : BK;
      const unsigned kstride = local ? BK : A_BYTES;
      const __amdgpu_buffer_rsrc_t rsA = __builtin_amdgcn_make_buffer_rsrc((void*)abase, 0, BM * K, 0x00020000);
      const __amdgpu_buffer_rsrc_t rsS = __builtin_amdgcn_make_buffer_rsrc((void*)(myws + WS_STG + (size_t)(par * NR + src) * STG_SRC + (size_t)half * 4 * A_BYTES), 0, 4 * A_BYTES, 0x00020000);
      const unsigned vA0 = lq * rowstride + ((lc ^ (lane >> 4)) * 16);
      const unsigned vA1 = lq * rowstride + ((lc ^ (4 + (lane >> 4))) * 16);
      int nj = 0, nkt = local ? 0 : NPUSH, npi = 0, npp = (local || NPUSH == 0) ? 1000 : pos0;
      unsigned islot = 0;
#define DMA_A(rs, slotoff, i, so) __builtin_amdgcn_raw_ptr_buffer_load_lds(rs, (LAS void*)(lds + (slotoff) + (i) * 1024), 16, ((i) & 1) ? vA1 : vA0, (so) + (i) * 8 * rowstride, 0, 0)
#define ISSUE() do { \
        int kt_; bool ps_ = false; \
        if (nj == npp) { kt_ = npi; ps_ = true; ++npi; npp = (npi < NPUSH) ? npp + PSTR : 1000; } else { kt_ = nkt; ++nkt; } \
        if (hw == 3) { _Pragma("unroll") for (int j = 0; j < 16; ++j) DMA_B(islot + A_BYTES, j, kt_); } \
        else { \
          const unsigned lo_ = islot + (hw - 1) * 16384, so_ = kt_ * kstride + (hw - 1) * 128 * rowstride; \
          if (ps_) { _Pragma("unroll") for (int i = 0; i < 16; ++i) DMA_A(rsS, lo_, i, so_); } \
          else { _Pragma("unroll") for (int i = 0; i < 16; ++i) DMA_A(rsA, lo_, i, so_); } \
        } \
        ++nj; islot += SLOT; if (islot == NSLOT * SLOT) islot = 0; } while (0)
#define ISSUE_G(gg) do { \
          const unsigned lo_ = g_off(gg) + (hw - 1) * 4096; \
          const unsigned so_ = local ? (NE + ((gg) & 3)) * BK + ((hw - 1) * 128 + ((gg) >> 2) * 32) * K : NE * A_BYTES + ((gg) * 2 + (hw - 1)) * 4096; \
          _Pragma("unroll") for (int i = 0; i < 4; ++i) DMA_A(rsA, lo_, i, so_); } while (0)
      if (hw == 3) { ISSUE(); ISSUE(); ISSUE(); }
      BAR();
      if (hw != 3) { ISSUE(); ISSUE(); ISSUE(); }
      WAIT_VM(32);
      BAR();
      for (int js = 0; js < NE - 3; ++js) {
        WAIT_VM(16);
        BAR();
        ISSUE();
      }
      if (hw != 3) {
#define GSTEP(sidx) do { WAIT_VM((sidx) == -3 ? 16 : g_allow(sidx)); BAR(); \
          _Pragma("unroll") for (int gg = 0; gg < 16; ++gg) if (g_issue(gg) == (sidx)) ISSUE_G(gg); } while (0)
        GSTEP(-3); GSTEP(-2); GSTEP(-1);
        GSTEP(0); GSTEP(1); GSTEP(2); GSTEP(3); GSTEP(4); GSTEP(5); GSTEP(6); GSTEP(7);
        GSTEP(8); GSTEP(9); GSTEP(10); GSTEP(11); GSTEP(12); GSTEP(13); GSTEP(14);
        BAR();
      } else {
        WAIT_VM(16); BAR(); ISSUE_B(1, 0, 16); ISSUE_B(2, 0, 16);
        WAIT_VM(32); BAR(); ISSUE_B(3, 8, 16);
        unsigned tv2 = __hip_atomic_load(tmo, __ATOMIC_RELAXED, __HIP_MEMORY_SCOPE_AGENT);
        BAR();
        WAIT_VM(25); BAR();
        WAIT_VM(9); BAR();
        WAIT_VM(0); BAR();
        asm volatile("" : "+v"(tv2));
        s = (tv2 == call) ? __builtin_nanf("") : sc;
        BAR(); BARM();
        DRAIN(0, 1, true); DRAIN(1, 1, true); DRAIN(2, 1, false);
      }
    }
    BARM();
    if (hw == 0 || hw == 3) {
      const int sw = hw & 1;
      const unsigned lb = sw ? st_off(3, 1) : st_off(3, 0);
      const unsigned gbo = (sw * 128 + 3 * 32) * (NS * 4);
#pragma unroll
      for (int i = 0; i < 16; ++i) d[i] = *(const LAS v4f*)(lds + lb + i * 1024 + (vL ^ (((2 * i) & 15) * 16)));
#pragma unroll
      for (int i = 0; i < 16; ++i) { d[i] = d[i] * s; asm volatile("" : "+v"(d[i])); }
      SCHED();
#pragma unroll
      for (int i = 0; i < 16; ++i) __builtin_amdgcn_raw_buffer_store_b128(__builtin_bit_cast(u32x4, d[i]), rsC, vG, gbo + 2 * i * (NS * 4), 16);
      STNOP();
    }
  }
  if (tid == 256 && (ticket % NWG) == NWG - 1)
    __hip_atomic_store((unsigned*)(myws + WS_CTR), call, __ATOMIC_RELAXED, __HIP_MEMORY_SCOPE_AGENT);
}

}

extern "C" void kernel_launch(int rank, int nranks, ncclComm_t comm,
                              hipStream_t stream,
                              void* const* d_in, const int* in_sizes, int n_in,
                              void* d_out, int out_size,
                              const int* in_dtypes, int out_dtype,
                              void* const* d_out_all,
                              void* const* d_ws_all, size_t ws_bytes) {
  if (nranks != NR || n_in < 4) { fprintf(stderr, "unsupported configuration\n"); return; }
  Args a;
  a.x = (const unsigned char*)d_in[0];
  a.W = (const unsigned char*)d_in[1];
  a.sx = (const float*)d_in[2];
  a.sw = (const float*)d_in[3];
  a.out = (float*)d_out;
  for (int p = 0; p < NR; ++p) a.wsr[p] = (unsigned char*)d_ws_all[(rank + p) & 7];
  a.rank = rank;
  hipLaunchKernelGGL(fused_gemm, dim3(NWG), dim3(512), 0, stream, a);
  HIP_CHECK(hipGetLastError());
}
```

```cpp
#include <hip/hip_runtime.h>
#include <rccl/rccl.h>
#include <cstdio>

#define HIP_CHECK(expr) do { hipError_t _e = (expr); if (_e != hipSuccess) { \
    fprintf(stderr, "HIP error %s at %s:%d\n", hipGetErrorString(_e), __FILE__, __LINE__); \
    return; } } while (0)

namespace {

typedef int v8i __attribute__((ext_vector_type(8)));
typedef int v4i __attribute__((ext_vector_type(4)));
typedef int v2i __attribute__((ext_vector_type(2)));
typedef float v4f __attribute__((ext_vector_type(4)));
typedef unsigned u32x4 __attribute__((ext_vector_type(4)));
#define LAS __attribute__((address_space(3)))

constexpr int NR = 8;
constexpr int M = 4096, K = 4096, NW = 16384, NS = 2048, MS = 512;
constexpr int BM = 256, BN = 128, BK = 128, NT = K / BK;
constexpr int A_BYTES = BM * BK, B_BYTES = BK * BN, SLOT = A_BYTES + B_BYTES;
constexpr int NSLOT = 3;
constexpr int LDS_BYTES = NSLOT * SLOT + B_BYTES;
constexpr int NWG = 256;
#define NPUSHV 2
#define TD0 150
#define TD1 100
#define TDS 0
#define PBASE 6
constexpr int NPUSH = NPUSHV;
#define PSTRV (NPUSHV <= 2 ? 8 : NPUSHV == 3 ? 6 : 4)
constexpr int PSTR = PSTRV;

constexpr int NE = NT - 4;
static_assert(PBASE + 7 + (NPUSH - 1) * PSTR < NE, "pushed steps must lie in the ring part");
static_assert(NE % NSLOT == 1, "the region tables below assume that step NE-3 sits in slot 1");
__host__ __device__ constexpr unsigned g_off(int g) {
  return g < 2 ? 49152u + g * 8192u : g < 6 ? 98304u + (g - 2) * 8192u : g < 12 ? (g - 6) * 8192u : g < 14 ? 114688u + (g - 12) * 8192u : 49152u + (g - 14) * 8192u; }
__host__ __device__ constexpr unsigned gb_off(int j) { return j == 0 ? 147456u : j == 1 ? 65536u : j == 2 ? 81920u : 131072u; }
#define GD 4
__host__ __device__ constexpr int g_avail(int g) { return g < 2 ? -3 : g < 6 ? -2 : g < 12 ? -1 : g == 12 ? 4 : 5; }
#define GD2 GD
#define GSW 8
__host__ __device__ constexpr int g_issue(int g) {
  int m = -3;
  for (int q = 0; q <= g; ++q) { const int d = q - 1 - (q < GSW ? GD : GD2); const int v = g_avail(q) > d ? g_avail(q) : d; if (v > m) m = v; }
  return m; }
__host__ __device__ constexpr int g_allow(int sidx) {
  int n = 0;
  for (int g = 0; g < 16; ++g) if (g_issue(g) < sidx && g > sidx + 1) ++n;
  return 4 * n; }
__host__ __device__ constexpr unsigned st_off(int p, int w) {
  return p == 0 ? (w ? 98304u : 49152u) : p == 2 ? (w ? 32768u : 16384u) : (w ? 0u : 98304u); }

constexpr size_t WS_CTR = 0, WS_TICK = 128, WS_TMO = 256;
constexpr size_t WS_FLAGE = 65536;
constexpr size_t WS_FLAGP = 65536 + 16384;
constexpr size_t WS_STAMP = 131072;
constexpr size_t WS_EXP = 1u << 20;
constexpr size_t WS_STG = 8u << 20;
constexpr size_t SHARD = (size_t)MS * K;
constexpr unsigned STG_SRC = 2 * 4 * A_BYTES;
constexpr unsigned long long DEADLINE_TICKS = 50000000ull;

struct Args {
  const unsigned char* x; const unsigned char* W; const float* sx; const float* sw; float* out;
  unsigned char* wsr[NR];
  int rank;
};

#define BAR() __builtin_amdgcn_s_barrier()
#define DBGNOP() do {} while (0)
#define SCHED() __builtin_amdgcn_sched_barrier(0)
#define WAIT_LGKM0() do { asm volatile("" ::: "memory"); __builtin_amdgcn_s_waitcnt(0xC07F); asm volatile("" ::: "memory"); } while (0)
#define BARM() do { asm volatile("" ::: "memory"); __builtin_amdgcn_s_barrier(); asm volatile("" ::: "memory"); } while (0)
#define WAIT_VM(n) asm volatile("s_waitcnt vmcnt(%0)" :: "n"(n) : "memory")

__device__ __forceinline__ v8i mk8(v4i a, v4i b) { return __builtin_shufflevector(a, b, 0, 1, 2, 3, 4, 5, 6, 7); }

__global__ void __launch_bounds__(512, 2) fused_gemm(Args a) {
  __shared__ __attribute__((aligned(1024))) unsigned char smem[LDS_BYTES];
  LAS unsigned char* lds = (LAS unsigned char*)smem;
  const int tid = threadIdx.x, lane = tid & 63, wid = __builtin_amdgcn_readfirstlane(tid >> 6);
  const int b = blockIdx.x;
  const int xl = b & 7, half = (b >> 3) & 1, tn = b >> 4;
  const int src = (a.rank + xl) & 7;
  const int tm = src * 2 + half;
  unsigned char* const myws = a.wsr[0];

  const unsigned craw = __hip_atomic_load((unsigned*)(myws + WS_CTR), __ATOMIC_RELAXED, __HIP_MEMORY_SCOPE_AGENT);
  const unsigned long long tstart = __builtin_amdgcn_s_memrealtime();
  unsigned call;
  const bool exporter = (xl < 4);
  const int rw = (b >> 3) * 4 + (b & 3);
  unsigned char* ppeer = nullptr; unsigned* pflag = nullptr;
  if (exporter) {
    const int h = rw >> 6, r0 = (rw & 63) * 4;
    const __amdgpu_buffer_rsrc_t rs = __builtin_amdgcn_make_buffer_rsrc((void*)(a.x + (size_t)(h * 256 + r0) * K), 0, 4 * K, 0x00020000);
    u32x4 v0 = __builtin_amdgcn_raw_buffer_load_b128(rs, tid * 16, 0, 0);
    u32x4 v1 = __builtin_amdgcn_raw_buffer_load_b128(rs, tid * 16 + 8192, 0, 0);
    unsigned char* pl = a.wsr[lane & 7];
    call = __builtin_amdgcn_readfirstlane(craw) + 1u;
    const __amdgpu_buffer_rsrc_t rd = __builtin_amdgcn_make_buffer_rsrc((void*)(myws + WS_EXP + (call & 1u) * SHARD + (size_t)h * (SHARD / 2)), 0, (unsigned)(SHARD / 2), 0x00020000);
    const int o = tid * 16;
    const int row = r0 + (o >> 12), kt = (o & 4095) >> 7, c = o & 127;
#define EXP_OFF(row_) (kt < NE ? kt * A_BYTES + (row_) * BK : NE * A_BYTES + (((((row_) >> 5) & 3) * 4 + (kt - NE)) * 2 + ((row_) >> 7)) * 4096 + ((row_) & 31) * BK)
    __builtin_amdgcn_raw_buffer_store_b128(v0, rd, EXP_OFF(row) + c, 0, 17);
    __builtin_amdgcn_raw_buffer_store_b128(v1, rd, EXP_OFF(row + 2) + c, 0, 17);
    asm volatile("s_waitcnt vmcnt(0)" ::: "memory");
    BAR();
    if (wid == 4 && lane < NR && lane != 0)
      __hip_atomic_store((unsigned*)(pl + WS_FLAGE) + a.rank * 128 + rw, call, __ATOMIC_RELAXED, __HIP_MEMORY_SCOPE_SYSTEM);
  } else if (wid < 4 && NPUSH > 0) {
    const int u = rw * 4 + wid;
    if (u < 64 * 7) {
      const int pc = u / 7, po = 1 + u % 7, ph = pc >> 5, pi = pc & 31;
      const __amdgpu_buffer_rsrc_t rsX = __builtin_amdgcn_make_buffer_rsrc((void*)(a.x + (size_t)(ph * 256 + pi * 8) * K), 0, 8 * K, 0x00020000);
      const unsigned vX = (lane >> 3) * K + (lane & 7) * 16;
      u32x4 v[NPUSH > 0 ? NPUSH : 1];
#pragma unroll
      for (int k = 0; k < NPUSH; ++k) v[k] = __builtin_amdgcn_raw_buffer_load_b128(rsX, vX, k * BK, 0);
      ppeer = a.wsr[po];
      call = __builtin_amdgcn_readfirstlane(craw) + 1u;
      const size_t so = WS_STG + (size_t)((call & 1u) * NR + a.rank) * STG_SRC + (size_t)(ph * 4) * A_BYTES + pi * 1024;
      const __amdgpu_buffer_rsrc_t rd = __builtin_amdgcn_make_buffer_rsrc((void*)(ppeer + so), 0, 4 * A_BYTES, 0x00020000);
#pragma unroll
      for (int k = 0; k < NPUSH; ++k) {
        if (k > 0 && TD0 > 0) { while (__builtin_amdgcn_s_memrealtime() - tstart < (unsigned long long)(TD0 + (k - 1) * TD1)) __builtin_amdgcn_s_sleep(1); }
        else if (k == 0 && TDS > 0) { while (__builtin_amdgcn_s_memrealtime() - tstart < (unsigned long long)(TDS * (u & 3))) __builtin_amdgcn_s_sleep(1); }
        __builtin_amdgcn_raw_buffer_store_b128(v[k], rd, lane * 16, k * A_BYTES, 17);
      }
      pflag = (unsigned*)(ppeer + WS_FLAGP) + a.rank * 64 + pc;
    }
  }
  call = __builtin_amdgcn_readfirstlane(craw) + 1u;
  const unsigned par = call & 1u;
  unsigned ticket = 0;

  if (wid < 4) {
    const int wm = wid >> 1, wn = wid & 1;
    const int r = lane & 15, g = lane >> 4;
    const unsigned aoff0 = (wm * 128 + r) * 128 + ((g ^ ((r >> 1) & 7)) * 16);
    const unsigned aoff1 = aoff0 ^ 64;
    const int q = r >> 1, p = r & 1;
    unsigned boff[4];
#pragma unroll
    for (int nn = 0; nn < 4; ++nn) boff[nn] = A_BYTES + 2 * g * 1024 + q * 128 + (((wn * 4 + nn) ^ ((q >> 1) | ((g & 1) << 2))) * 16) + p * 8;
    v4f acc[8][4];
#pragma unroll
    for (int m = 0; m < 8; ++m)
#pragma unroll
      for (int nn = 0; nn < 4; ++nn) acc[m][nn] = (v4f){0.f, 0.f, 0.f, 0.f};
    v8i Bf[2][4];
    v8i Af[4];
#define RDA(slotoff, m) mk8(*(const LAS v4i*)(lds + (slotoff) + aoff0 + (m) * 2048), *(const LAS v4i*)(lds + (slotoff) + aoff1 + (m) * 2048))
#define RDB2(slotoff, nn, r_) __builtin_amdgcn_ds_read_tr8_b64_v2i32((LAS v2i*)(lds + (slotoff) + boff[nn] + ((r_) & 1) * 1024 + ((r_) >> 1) * 8192))
#define RDB(slotoff, nn) ({ v2i b0 = RDB2(slotoff, nn, 0), b1 = RDB2(slotoff, nn, 1), b2 = RDB2(slotoff, nn, 2), b3 = RDB2(slotoff, nn, 3); \
      v4i lo = __builtin_shufflevector(b0, b1, 0, 1, 2, 3), hi = __builtin_shufflevector(b2, b3, 0, 1, 2, 3); mk8(lo, hi); })
#define MMA4(m, Bc, Ac) do { _Pragma("unroll") for (int nn = 0; nn < 4; ++nn) \
      acc[m][nn] = __builtin_amdgcn_mfma_scale_f32_16x16x128_f8f6f4(Bc[nn], Ac, acc[m][nn], 1, 0, 0, 0, 0, 0); } while (0)
    BAR();
    if (pflag) {
      asm volatile("s_waitcnt vmcnt(0)" ::: "memory");
      if (lane == 0) __hip_atomic_store(pflag, call, __ATOMIC_RELAXED, __HIP_MEMORY_SCOPE_SYSTEM);
    }
    BAR();
    {
      const unsigned so = 0;
#pragma unroll
      for (int nn = 0; nn < 4; ++nn) Bf[0][nn] = RDB(so, nn);
      Af[0] = RDA(so, 0); Af[1] = RDA(so, 1);
    }
    unsigned so = 0;
#define KSTEP(cur, nxt, ADJ, BADJ) do { \
      const unsigned sn = (so == 2 * SLOT) ? 0u : so + SLOT; \
      const unsigned sa = sn - (ADJ), sb = sn + (BADJ); \
      Af[2] = RDA(so, 2); SCHED(); MMA4(0, Bf[cur], Af[0]); SCHED(); \
      Af[3] = RDA(so, 3); SCHED(); MMA4(1, Bf[cur], Af[1]); SCHED(); \
      Af[0] = RDA(so, 4); SCHED(); MMA4(2, Bf[cur], Af[2]); SCHED(); \
      Af[1] = RDA(so, 5); SCHED(); MMA4(3, Bf[cur], Af[3]); SCHED(); \
      Af[2] = RDA(so, 6); Af[3] = RDA(so, 7); SCHED(); MMA4(4, Bf[cur], Af[0]); SCHED(); \
      MMA4(5, Bf[cur], Af[1]); SCHED(); \
      asm volatile("s_waitcnt lgkmcnt(0)" ::: "memory"); \
      BAR(); SCHED(); \
      Bf[nxt][0] = RDB(sb, 0); Bf[nxt][1] = RDB(sb, 1); Af[0] = RDA(sa, 0); SCHED(); \
      MMA4(6, Bf[cur], Af[2]); SCHED(); \
      Bf[nxt][2] = RDB(sb, 2); Bf[nxt][3] = RDB(sb, 3); Af[1] = RDA(sa, 1); SCHED(); \
      MMA4(7, Bf[cur], Af[3]); SCHED(); \
      so = sn; } while (0)
    static_assert(g_off(0) == (NE % NSLOT) * SLOT, "granule 0 must sit in the ring slot of step NE");
    for (int kt = 0; kt < NE; kt += 2) {
      KSTEP(0, 1, 0u, 0u);
      KSTEP(1, 0, (kt == NE - 2) ? wm * 12288u : 0u, (kt == NE - 2) ? gb_off(0) - (g_off(0) + A_BYTES) : 0u);
    }
    unsigned bq[4], ga0 = aoff0 - wm * 12288u + g_off(1), ga1 = ga0 ^ 64u;
#pragma unroll
    for (int nn = 0; nn < 4; ++nn) { bq[nn] = boff[nn] + (gb_off(1) - A_BYTES); asm volatile("" : "+v"(bq[nn])); }
    asm volatile("" : "+v"(ga0), "+v"(ga1));
    const unsigned d0 = r * 512 + wn * 256 + ((g ^ r) * 16);
#define GRDA(mm) mk8(*(const LAS v4i*)(lds + ga0 + (mm) * 2048), *(const LAS v4i*)(lds + ga1 + (mm) * 2048))
#define GRDB2(nn, r_) __builtin_amdgcn_ds_read_tr8_b64_v2i32((LAS v2i*)(lds + bq[nn] + ((r_) & 1) * 1024 + ((r_) >> 1) * 8192))
#define GRDB(nn) ({ v2i b0 = GRDB2(nn, 0), b1 = GRDB2(nn, 1), b2 = GRDB2(nn, 2), b3 = GRDB2(nn, 3); \
      v4i lo = __builtin_shufflevector(b0, b1, 0, 1, 2, 3), hi = __builtin_shufflevector(b2, b3, 0, 1, 2, 3); mk8(lo, hi); })
#define MM1(m, nn, Bc, Ac) do { acc[m][nn] = __builtin_amdgcn_mfma_scale_f32_16x16x128_f8f6f4(Bc[nn], Ac, acc[m][nn], 1, 0, 0, 0, 0, 0); SCHED(); } while (0)
#define PSTEP(s) do { \
      WAIT_LGKM0(); BARM(); SCHED(); \
      MM1(((s) >> 2) * 2, 0, Bf[(s) & 1], Af[((s) & 1) * 2]); \
      if ((s) < 15) { Bf[((s) + 1) & 1][0] = GRDB(0); SCHED(); } \
      MM1(((s) >> 2) * 2, 1, Bf[(s) & 1], Af[((s) & 1) * 2]); \
      if ((s) < 15) { Af[(((s) + 1) & 1) * 2] = GRDA(0); SCHED(); } \
      MM1(((s) >> 2) * 2, 2, Bf[(s) & 1], Af[((s) & 1) * 2]); \
      if ((s) < 15) { Bf[((s) + 1) & 1][1] = GRDB(1); SCHED(); } \
      MM1(((s) >> 2) * 2, 3, Bf[(s) & 1], Af[((s) & 1) * 2]); \
      if ((s) < 15) { Bf[((s) + 1) & 1][2] = GRDB(2); SCHED(); } \
      MM1(((s) >> 2) * 2 + 1, 0, Bf[(s) & 1], Af[((s) & 1) * 2 + 1]); \
      if ((s) < 15) { Bf[((s) + 1) & 1][3] = GRDB(3); SCHED(); } \
      MM1(((s) >> 2) * 2 + 1, 1, Bf[(s) & 1], Af[((s) & 1) * 2 + 1]); \
      if ((s) < 15) { Af[(((s) + 1) & 1) * 2 + 1] = GRDA(1); SCHED(); } \
      MM1(((s) >> 2) * 2 + 1, 2, Bf[(s) & 1], Af[((s) & 1) * 2 + 1]); \
      if ((s) < 15) { _Pragma("unroll") for (int nn = 0; nn < 4; ++nn) { bq[nn] += gb_off(((s) + 2) & 3) - gb_off(((s) + 1) & 3); asm volatile("" : "+v"(bq[nn])); } \
        ga0 += g_off((s) < 14 ? (s) + 2 : 15) - g_off((s) + 1); ga1 += g_off((s) < 14 ? (s) + 2 : 15) - g_off((s) + 1); asm volatile("" : "+v"(ga0), "+v"(ga1)); SCHED(); } \
      MM1(((s) >> 2) * 2 + 1, 3, Bf[(s) & 1], Af[((s) & 1) * 2 + 1]); \
      if (((s) & 3) == 3) { unsigned db = d0 + (wm ? st_off((s) >> 2, 1) : st_off((s) >> 2, 0)); asm volatile("" : "+v"(db)); \
        _Pragma("unroll") for (int nn = 0; nn < 4; ++nn) _Pragma("unroll") for (int mm = 0; mm < 2; ++mm) \
          *(LAS v4f*)(lds + (db ^ (nn * 64)) + mm * 8192) = acc[((s) >> 2) * 2 + mm][nn]; SCHED(); } \
      } while (0)
    PSTEP(0); PSTEP(1); PSTEP(2); PSTEP(3);
    PSTEP(4); PSTEP(5); PSTEP(6); PSTEP(7);
    PSTEP(8); PSTEP(9); PSTEP(10); PSTEP(11);
    PSTEP(12); PSTEP(13); PSTEP(14);
    PSTEP(15);
    WAIT_LGKM0(); BARM();
  } else {
    const int hw = wid - 4;
    const bool local = (src == a.rank);
    const int pos0 = PBASE + (tn & 7);
    unsigned* const tmo = (unsigned*)(myws + WS_TMO);
    const __amdgpu_buffer_rsrc_t rsC = __builtin_amdgcn_make_buffer_rsrc((void*)(a.out + (size_t)(tm * BM) * NS + tn * BN), 0, BM * NS * 4, 0x00020000);
    const unsigned vG = (lane >> 5) * (NS * 4) + (lane & 31) * 16;
    const unsigned vL = (lane >> 5) * 512 + (((lane & 31) ^ (lane >> 5)) * 16);
    const float sc = a.sx[0] * a.sw[0];
    float s;
    v4f d[16];
    const __amdgpu_buffer_rsrc_t rsB = __builtin_amdgcn_make_buffer_rsrc((void*)(a.W + a.rank * NS + tn * BN), 0, (unsigned)((size_t)K * NW - (a.rank * NS + tn * BN)), 0x00020000);
    const int lq = lane >> 3, lc = lane & 7;
    const unsigned vB0 = lq * NW + ((lc ^ (lq >> 1)) * 16);
    const unsigned vB1 = lq * NW + ((lc ^ ((lq >> 1) | 4)) * 16);
#define DMA_B(boffs, j, kt) __builtin_amdgcn_raw_ptr_buffer_load_lds(rsB, (LAS void*)(lds + (boffs) + (j) * 1024), 16, (((j) >> 1) & 1) ? vB1 : vB0, ((kt) * BK + (j) * 8) * NW, 0, 0)
#define ISSUE_B(j, j0, j1) do { _Pragma("unroll") for (int jj = (j0); jj < (j1); ++jj) DMA_B(gb_off(j), jj, NE + (j)); } while (0)
#define ORD(pb, sw, i) (*(const LAS v4f*)(lds + st_off(pb, sw) + (i) * 1024 + (vL ^ (((2 * (i)) & 15) * 16))))
#define OST(pb, sw, i, v) __builtin_amdgcn_raw_buffer_store_b128(__builtin_bit_cast(u32x4, v), rsC, vG, ((sw) * 128 + (pb) * 32 + 2 * (i)) * (NS * 4), 16)
#define STNOP() do { SCHED(); asm volatile("s_nop 1"); SCHED(); } while (0)
#define DRAIN(pb, sw, LASTB) do { \
      _Pragma("unroll") for (int i = 0; i < 16; ++i) d[i] = ORD(pb, sw, i); \
      _Pragma("unroll") for (int i = 0; i < 16; ++i) { d[i] = d[i] * s; asm volatile("" : "+v"(d[i])); } \
      SCHED(); \
      _Pragma("unroll") for (int i = 0; i < 4; ++i) OST(pb, sw, i, d[i]); \
      STNOP(); WAIT_LGKM0(); BARM(); \
      _Pragma("unroll") for (int i = 4; i < 8; ++i) OST(pb, sw, i, d[i]); \
      STNOP(); BARM(); \
      _Pragma("unroll") for (int i = 8; i < 12; ++i) OST(pb, sw, i, d[i]); \
      STNOP(); BARM(); \
      _Pragma("unroll") for (int i = 12; i < 16; ++i) OST(pb, sw, i, d[i]); \
      STNOP(); \
      if (LASTB) BARM(); } while (0)
    if (hw == 0) {
      const unsigned long long t0 = __builtin_amdgcn_s_memrealtime();
#define WAIT_FLAGS(f) do { for (;;) { \
          const unsigned v_ = __hip_atomic_load(f, __ATOMIC_RELAXED, __HIP_MEMORY_SCOPE_SYSTEM); \
          const unsigned tv_ = __hip_atomic_load(tmo, __ATOMIC_RELAXED, __HIP_MEMORY_SCOPE_AGENT); \
          if (__all(v_ >= call)) break; \
          if (tv_ == call || __builtin_amdgcn_s_memrealtime() - t0 > DEADLINE_TICKS) { \
            if (lane == 0) __hip_atomic_store(tmo, call, __ATOMIC_RELAXED, __HIP_MEMORY_SCOPE_AGENT); \
            asm volatile("s_waitcnt vmcnt(0)" ::: "memory"); \
            break; \
          } } } while (0)
#define WAIT_FLAGS2(f) do { for (;;) { \
          const unsigned v_ = __hip_atomic_load(f, __ATOMIC_RELAXED, __HIP_MEMORY_SCOPE_SYSTEM); \
          const unsigned w_ = __hip_atomic_load(f + 64, __ATOMIC_RELAXED, __HIP_MEMORY_SCOPE_SYSTEM); \
          const unsigned tv_ = __hip_atomic_load(tmo, __ATOMIC_RELAXED, __HIP_MEMORY_SCOPE_AGENT); \
          if (__all(v_ >= call && w_ >= call)) break; \
          if (tv_ == call || __builtin_amdgcn_s_memrealtime() - t0 > DEADLINE_TICKS) { \
            if (lane == 0) __hip_atomic_store(tmo, call, __ATOMIC_RELAXED, __HIP_MEMORY_SCOPE_AGENT); \
            asm volatile("s_waitcnt vmcnt(0)" ::: "memory"); \
            break; \
          } } } while (0)
      const unsigned* fE = (const unsigned*)(myws + WS_FLAGE) + src * 128 + lane;
      const unsigned* fP = (const unsigned*)(myws + WS_FLAGP) + src * 64 + half * 32 + (lane & 31);
      if (!local) {
        WAIT_FLAGS2(fE);
      }
      BAR();
      if (lane == 0) ticket = __hip_atomic_fetch_add((unsigned*)(myws + WS_TICK), 1u, __ATOMIC_RELAXED, __HIP_MEMORY_SCOPE_AGENT);
      BAR();
      const int JW = pos0 - 3;
      int js = 0;
      if (!local && NPUSH > 0) {
        bool seen = false;
        while (js + 2 <= JW) {
          const unsigned vP = __hip_atomic_load(fP, __ATOMIC_RELAXED, __HIP_MEMORY_SCOPE_SYSTEM);
          BAR();
          seen = __all(vP >= call);
          BAR();
          js += 2;
          if (seen) break;
        }
        if (!seen) {
          for (; js < JW; ++js) BAR();
          WAIT_FLAGS(fP);
        }
      }
      for (; js < NE - 3; ++js) BAR();
      ISSUE_B(0, 0, 16);
      BAR(); BAR();
      ISSUE_B(3, 0, 8);
      unsigned tv2 = __hip_atomic_load(tmo, __ATOMIC_RELAXED, __HIP_MEMORY_SCOPE_AGENT);
      WAIT_VM(9); BAR();
      BAR(); BAR();
      WAIT_VM(0); BAR();
      asm volatile("" : "+v"(tv2));
      s = (tv2 == call) ? __builtin_nanf("") : sc;
      BAR(); BARM();
      DRAIN(0, 0, true); DRAIN(1, 0, true); DRAIN(2, 0, false);
    } else {
      const unsigned char* abase = local ? (a.x + (size_t)half * BM * K) : (a.wsr[xl] + WS_EXP + par * SHARD + (size_t)half * (SHARD / 2));
      const unsigned rowstride = local ? K : BK;
      const unsigned kstride = local ? BK : A_BYTES;
      const __amdgpu_buffer_rsrc_t rsA = __builtin_amdgcn_make_buffer_rsrc((void*)abase, 0, BM * K, 0x00020000);
      const __amdgpu_buffer_rsrc_t rsS = __builtin_amdgcn_make_buffer_rsrc((void*)(myws + WS_STG + (size_t)(par * NR + src) * STG_SRC + (size_t)half * 4 * A_BYTES), 0, 4 * A_BYTES, 0x00020000);
      const unsigned vA0 = lq * rowstride + ((lc ^ (lane >> 4)) * 16);
      const unsigned vA1 = lq * rowstride + ((lc ^ (4 + (lane >> 4))) * 16);
      int nj = 0, nkt = local ? 0 : NPUSH, npi = 0, npp = (local || NPUSH == 0) ? 1000 : pos0;
      unsigned islot = 0;
#define DMA_A(rs, slotoff, i, so) __builtin_amdgcn_raw_ptr_buffer_load_lds(rs, (LAS void*)(lds + (slotoff) + (i) * 1024), 16, ((i) & 1) ? vA1 : vA0, (so) + (i) * 8 * rowstride, 0, 0)
#define ISSUE() do { \
        int kt_; bool ps_ = false; \
        if (nj == npp) { kt_ = npi; ps_ = true; ++npi; npp = (npi < NPUSH) ? npp + PSTR : 1000; } else { kt_ = nkt; ++nkt; } \
        if (hw == 3) { _Pragma("unroll") for (int j = 0; j < 16; ++j) DMA_B(islot + A_BYTES, j, kt_); } \
        else { \
          const unsigned lo_ = islot + (hw - 1) * 16384, so_ = kt_ * kstride + (hw - 1) * 128 * rowstride; \
          if (ps_) { _Pragma("unroll") for (int i = 0; i < 16; ++i) DMA_A(rsS, lo_, i, so_); } \
          else { _Pragma("unroll") for (int i = 0; i < 16; ++i) DMA_A(rsA, lo_, i, so_); } \
        } \
        ++nj; islot += SLOT; if (islot == NSLOT * SLOT) islot = 0; } while (0)
#define ISSUE_G(gg) do { \
          const unsigned lo_ = g_off(gg) + (hw - 1) * 4096; \
          const unsigned so_ = local ? (NE + ((gg) & 3)) * BK + ((hw - 1) * 128 + ((gg) >> 2) * 32) * K : NE * A_BYTES + ((gg) * 2 + (hw - 1)) * 4096; \
          _Pragma("unroll") for (int i = 0; i < 4; ++i) DMA_A(rsA, lo_, i, so_); } while (0)
      if (hw == 3) { ISSUE(); ISSUE(); ISSUE(); }
      BAR();
      if (hw != 3) { ISSUE(); ISSUE(); ISSUE(); }
      WAIT_VM(32);
      BAR();
      for (int js = 0; js < NE - 3; ++js) {
        WAIT_VM(16);
        BAR();
        ISSUE();
      }
      if (hw != 3) {
#define GSTEP(sidx) do { WAIT_VM((sidx) == -3 ? 16 : g_allow(sidx)); BAR(); \
          _Pragma("unroll") for (int gg = 0; gg < 16; ++gg) if (g_issue(gg) == (sidx)) ISSUE_G(gg); } while (0)
        GSTEP(-3); GSTEP(-2); GSTEP(-1);
        GSTEP(0); GSTEP(1); GSTEP(2); GSTEP(3); GSTEP(4); GSTEP(5); GSTEP(6); GSTEP(7);
        GSTEP(8); GSTEP(9); GSTEP(10); GSTEP(11); GSTEP(12); GSTEP(13); GSTEP(14);
        BAR();
      } else {
        WAIT_VM(16); BAR(); ISSUE_B(1, 0, 16); ISSUE_B(2, 0, 16);
        WAIT_VM(32); BAR(); ISSUE_B(3, 8, 16);
        unsigned tv2 = __hip_atomic_load(tmo, __ATOMIC_RELAXED, __HIP_MEMORY_SCOPE_AGENT);
        BAR();
        WAIT_VM(25); BAR();
        WAIT_VM(9); BAR();
        WAIT_VM(0); BAR();
        asm volatile("" : "+v"(tv2));
        s = (tv2 == call) ? __builtin_nanf("") : sc;
        BAR(); BARM();
        DRAIN(0, 1, true); DRAIN(1, 1, true); DRAIN(2, 1, false);
      }
    }
    BARM();
    if (hw == 0 || hw == 3) {
      const int sw = hw & 1;
      const unsigned lb = sw ? st_off(3, 1) : st_off(3, 0);
      const unsigned gbo = (sw * 128 + 3 * 32) * (NS * 4);
#pragma unroll
      for (int i = 0; i < 16; ++i) d[i] = *(const LAS v4f*)(lds + lb + i * 1024 + (vL ^ (((2 * i) & 15) * 16)));
#pragma unroll
      for (int i = 0; i < 16; ++i) { d[i] = d[i] * s; asm volatile("" : "+v"(d[i])); }
      SCHED();
#pragma unroll
      for (int i = 0; i < 16; ++i) __builtin_amdgcn_raw_buffer_store_b128(__builtin_bit_cast(u32x4, d[i]), rsC, vG, gbo + 2 * i * (NS * 4), 16);
      STNOP();
    }
  }
  if (tid == 256 && (ticket % NWG) == NWG - 1)
    __hip_atomic_store((unsigned*)(myws + WS_CTR), call, __ATOMIC_RELAXED, __HIP_MEMORY_SCOPE_AGENT);
}

}

extern "C" void kernel_launch(int rank, int nranks, ncclComm_t comm,
                              hipStream_t stream,
                              void* const* d_in, const int* in_sizes, int n_in,
                              void* d_out, int out_size,
                              const int* in_dtypes, int out_dtype,
                              void* const* d_out_all,
                              void* const* d_ws_all, size_t ws_bytes) {
  if (nranks != NR || n_in < 4) { fprintf(stderr, "unsupported configuration\n"); return; }
  Args a;
  a.x = (const unsigned char*)d_in[0];
  a.W = (const unsigned char*)d_in[1];
  a.sx = (const float*)d_in[2];
  a.sw = (const float*)d_in[3];
  a.out = (float*)d_out;
  for (int p = 0; p < NR; ++p) a.wsr[p] = (unsigned char*)d_ws_all[(rank + p) & 7];
  a.rank = rank;
  hipLaunchKernelGGL(fused_gemm, dim3(NWG), dim3(512), 0, stream, a);
  HIP_CHECK(hipGetLastError());
}
```

```cpp
#include <hip/hip_runtime.h>
#include <rccl/rccl.h>
#include <cstdio>

#define HIP_CHECK(expr) do { hipError_t _e = (expr); if (_e != hipSuccess) { \
    fprintf(stderr, "HIP error %s at %s:%d\n", hipGetErrorString(_e), __FILE__, __LINE__); \
    return; } } while (0)

namespace {

typedef int v8i __attribute__((ext_vector_type(8)));
typedef int v4i __attribute__((ext_vector_type(4)));
typedef int v2i __attribute__((ext_vector_type(2)));
typedef float v4f __attribute__((ext_vector_type(4)));
typedef unsigned u32x4 __attribute__((ext_vector_type(4)));
#define LAS __attribute__((address_space(3)))

constexpr int NR = 8;
constexpr int M = 4096, K = 4096, NW = 16384, NS = 2048, MS = 512;
constexpr int BM = 256, BN = 128, BK = 128, NT = K / BK;
constexpr int A_BYTES = BM * BK, B_BYTES = BK * BN, SLOT = A_BYTES + B_BYTES;
constexpr int NSLOT = 3;
constexpr int LDS_BYTES = NSLOT * SLOT + B_BYTES;
constexpr int NWG = 256;
#define NPUSHV 2
#define TD0 150
#define TD1 100
#define TDS 0
#define PBASE 6
constexpr int NPUSH = NPUSHV;
#define PSTRV (NPUSHV <= 2 ? 8 : NPUSHV == 3 ? 6 : 4)
constexpr int PSTR = PSTRV;

constexpr int NE = NT - 4;
static_assert(PBASE + 7 + (NPUSH - 1) * PSTR < NE, "pushed steps must lie in the ring part");
static_assert(NE % NSLOT == 1, "the region tables below assume that step NE-3 sits in slot 1");
__host__ __device__ constexpr unsigned g_off(int g) {
  return g < 2 ? 49152u + g * 8192u : g < 6 ? 98304u + (g - 2) * 8192u : g < 12 ? (g - 6) * 8192u : g < 14 ? 114688u + (g - 12) * 8192u : 49152u + (g - 14) * 8192u; }
__host__ __device__ constexpr unsigned gb_off(int j) { return j == 0 ? 147456u : j == 1 ? 65536u : j == 2 ? 81920u : 131072u; }
#define GD 4
__host__ __device__ constexpr int g_avail(int g) { return g < 2 ? -3 : g < 6 ? -2 : g < 12 ? -1 : g == 12 ? 4 : 5; }
#define GD2 GD
#define GSW 8
__host__ __device__ constexpr int g_issue(int g) {
  int m = -3;
  for (int q = 0; q <= g; ++q) { const int d = q - 1 - (q < GSW ? GD : GD2); const int v = g_avail(q) > d ? g_avail(q) : d; if (v > m) m = v; }
  return m; }
__host__ __device__ constexpr int g_allow(int sidx) {
  int n = 0;
  for (int g = 0; g < 16; ++g) if (g_issue(g) < sidx && g > sidx + 1) ++n;
  return 4 * n; }
__host__ __device__ constexpr unsigned st_off(int p, int w) {
  return p == 0 ? (w ? 98304u : 49152u) : p == 2 ? (w ? 32768u : 16384u) : (w ? 0u : 98304u); }

constexpr size_t WS_CTR = 0, WS_TICK = 128, WS_TMO = 256;
constexpr size_t WS_FLAGE = 65536;
constexpr size_t WS_FLAGP = 65536 + 16384;
constexpr size_t WS_STAMP = 131072;
constexpr size_t WS_EXP = 1u << 20;
constexpr size_t WS_STG = 8u << 20;
constexpr size_t SHARD = (size_t)MS * K;
constexpr unsigned STG_SRC = 2 * 4 * A_BYTES;
constexpr unsigned long long DEADLINE_TICKS = 50000000ull;

struct Args {
  const unsigned char* x; const unsigned char* W; const float* sx; const float* sw; float* out;
  unsigned char* wsr[NR];
  int rank;
  unsigned perm;
};

#define BAR() __builtin_amdgcn_s_barrier()
#define DBGNOP() do {} while (0)
#define SCHED() __builtin_amdgcn_sched_barrier(0)
#define WAIT_LGKM0() do { asm volatile("" ::: "memory"); __builtin_amdgcn_s_waitcnt(0xC07F); asm volatile("" ::: "memory"); } while (0)
#define BARM() do { asm volatile("" ::: "memory"); __builtin_amdgcn_s_barrier(); asm volatile("" ::: "memory"); } while (0)
#define WAIT_VM(n) asm volatile("s_waitcnt vmcnt(%0)" :: "n"(n) : "memory")

__device__ __forceinline__ v8i mk8(v4i a, v4i b) { return __builtin_shufflevector(a, b, 0, 1, 2, 3, 4, 5, 6, 7); }

__global__ void __launch_bounds__(512, 2) fused_gemm(Args a) {
  __shared__ __attribute__((aligned(1024))) unsigned char smem[LDS_BYTES];
  LAS unsigned char* lds = (LAS unsigned char*)smem;
  const int tid = threadIdx.x, lane = tid & 63, wid = __builtin_amdgcn_readfirstlane(tid >> 6);
  const int b = blockIdx.x;
  const int xl = b & 7, half = (b >> 3) & 1, tn = b >> 4;
  const int xo = (a.perm >> (4 * xl)) & 7;
  const int src = (a.rank + xo) & 7;
  const int tm = src * 2 + half;
  unsigned char* const myws = a.wsr[0];

  const unsigned craw = __hip_atomic_load((unsigned*)(myws + WS_CTR), __ATOMIC_RELAXED, __HIP_MEMORY_SCOPE_AGENT);
  const unsigned long long tstart = __builtin_amdgcn_s_memrealtime();
  unsigned call;
  const bool exporter = (xl < 4);
  const int rw = (b >> 3) * 4 + (b & 3);
  unsigned char* ppeer = nullptr; unsigned* pflag = nullptr;
  if (exporter) {
    const int h = rw >> 6, r0 = (rw & 63) * 4;
    const __amdgpu_buffer_rsrc_t rs = __builtin_amdgcn_make_buffer_rsrc((void*)(a.x + (size_t)(h * 256 + r0) * K), 0, 4 * K, 0x00020000);
    u32x4 v0 = __builtin_amdgcn_raw_buffer_load_b128(rs, tid * 16, 0, 0);
    u32x4 v1 = __builtin_amdgcn_raw_buffer_load_b128(rs, tid * 16 + 8192, 0, 0);
    unsigned char* pl = a.wsr[lane & 7];
    call = __builtin_amdgcn_readfirstlane(craw) + 1u;
    const __amdgpu_buffer_rsrc_t rd = __builtin_amdgcn_make_buffer_rsrc((void*)(myws + WS_EXP + (call & 1u) * SHARD + (size_t)h * (SHARD / 2)), 0, (unsigned)(SHARD / 2), 0x00020000);
    const int o = tid * 16;
    const int row = r0 + (o >> 12), kt = (o & 4095) >> 7, c = o & 127;
#define EXP_OFF(row_) (kt < NE ? kt * A_BYTES + (row_) * BK : NE * A_BYTES + (((((row_) >> 5) & 3) * 4 + (kt - NE)) * 2 + ((row_) >> 7)) * 4096 + ((row_) & 31) * BK)
    __builtin_amdgcn_raw_buffer_store_b128(v0, rd, EXP_OFF(row) + c, 0, 17);
    __builtin_amdgcn_raw_buffer_store_b128(v1, rd, EXP_OFF(row + 2) + c, 0, 17);
    asm volatile("s_waitcnt vmcnt(0)" ::: "memory");
    BAR();
    if (wid == 4 && lane < NR && lane != 0)
      __hip_atomic_store((unsigned*)(pl + WS_FLAGE) + a.rank * 128 + rw, call, __ATOMIC_RELAXED, __HIP_MEMORY_SCOPE_SYSTEM);
  } else if (wid < 4 && NPUSH > 0) {
    const int u = rw * 4 + wid;
    if (u < 64 * 7) {
      const int pc = u / 7, po = 1 + u % 7, ph = pc >> 5, pi = pc & 31;
      const __amdgpu_buffer_rsrc_t rsX = __builtin_amdgcn_make_buffer_rsrc((void*)(a.x + (size_t)(ph * 256 + pi * 8) * K), 0, 8 * K, 0x00020000);
      const unsigned vX = (lane >> 3) * K + (lane & 7) * 16;
      u32x4 v[NPUSH > 0 ? NPUSH : 1];
#pragma unroll
      for (int k = 0; k < NPUSH; ++k) v[k] = __builtin_amdgcn_raw_buffer_load_b128(rsX, vX, k * BK, 0);
      ppeer = a.wsr[po];
      call = __builtin_amdgcn_readfirstlane(craw) + 1u;
      const size_t so = WS_STG + (size_t)((call & 1u) * NR + a.rank) * STG_SRC + (size_t)(ph * 4) * A_BYTES + pi * 1024;
      const __amdgpu_buffer_rsrc_t rd = __builtin_amdgcn_make_buffer_rsrc((void*)(ppeer + so), 0, 4 * A_BYTES, 0x00020000);
#pragma unroll
      for (int k = 0; k < NPUSH; ++k) {
        if (k > 0 && TD0 > 0) { while (__builtin_amdgcn_s_memrealtime() - tstart < (unsigned long long)(TD0 + (k - 1) * TD1)) __builtin_amdgcn_s_sleep(1); }
        else if (k == 0 && TDS > 0) { while (__builtin_amdgcn_s_memrealtime() - tstart < (unsigned long long)(TDS * (u & 3))) __builtin_amdgcn_s_sleep(1); }
        __builtin_amdgcn_raw_buffer_store_b128(v[k], rd, lane * 16, k * A_BYTES, 17);
      }
      pflag = (unsigned*)(ppeer + WS_FLAGP) + a.rank * 64 + pc;
    }
  }
  call = __builtin_amdgcn_readfirstlane(craw) + 1u;
  const unsigned par = call & 1u;
  unsigned ticket = 0;

  if (wid < 4) {
    const int wm = wid >> 1, wn = wid & 1;
    const int r = lane & 15, g = lane >> 4;
    const unsigned aoff0 = (wm * 128 + r) * 128 + ((g ^ ((r >> 1) & 7)) * 16);
    const unsigned aoff1 = aoff0 ^ 64;
    const int q = r >> 1, p = r & 1;
    unsigned boff[4];
#pragma unroll
    for (int nn = 0; nn < 4; ++nn) boff[nn] = A_BYTES + 2 * g * 1024 + q * 128 + (((wn * 4 + nn) ^ ((q >> 1) | ((g & 1) << 2))) * 16) + p * 8;
    v4f acc[8][4];
#pragma unroll
    for (int m = 0; m < 8; ++m)
#pragma unroll
      for (int nn = 0; nn < 4; ++nn) acc[m][nn] = (v4f){0.f, 0.f, 0.f, 0.f};
    v8i Bf[2][4];
    v8i Af[4];
#define RDA(slotoff, m) mk8(*(const LAS v4i*)(lds + (slotoff) + aoff0 + (m) * 2048), *(const LAS v4i*)(lds + (slotoff) + aoff1 + (m) * 2048))
#define RDB2(slotoff, nn, r_) __builtin_amdgcn_ds_read_tr8_b64_v2i32((LAS v2i*)(lds + (slotoff) + boff[nn] + ((r_) & 1) * 1024 + ((r_) >> 1) * 8192))
#define RDB(slotoff, nn) ({ v2i b0 = RDB2(slotoff, nn, 0), b1 = RDB2(slotoff, nn, 1), b2 = RDB2(slotoff, nn, 2), b3 = RDB2(slotoff, nn, 3); \
      v4i lo = __builtin_shufflevector(b0, b1, 0, 1, 2, 3), hi = __builtin_shufflevector(b2, b3, 0, 1, 2, 3); mk8(lo, hi); })
#define MMA4(m, Bc, Ac) do { _Pragma("unroll") for (int nn = 0; nn < 4; ++nn) \
      acc[m][nn] = __builtin_amdgcn_mfma_scale_f32_16x16x128_f8f6f4(Bc[nn], Ac, acc[m][nn], 1, 0, 0, 0, 0, 0); } while (0)
    BAR();
    if (pflag) {
      asm volatile("s_waitcnt vmcnt(0)" ::: "memory");
      if (lane == 0) __hip_atomic_store(pflag, call, __ATOMIC_RELAXED, __HIP_MEMORY_SCOPE_SYSTEM);
    }
    BAR();
    {
      const unsigned so = 0;
#pragma unroll
      for (int nn = 0; nn < 4; ++nn) Bf[0][nn] = RDB(so, nn);
      Af[0] = RDA(so, 0); Af[1] = RDA(so, 1);
    }
    unsigned so = 0;
#define KSTEP(cur, nxt, ADJ, BADJ) do { \
      const unsigned sn = (so == 2 * SLOT) ? 0u : so + SLOT; \
      const unsigned sa = sn - (ADJ), sb = sn + (BADJ); \
      Af[2] = RDA(so, 2); SCHED(); MMA4(0, Bf[cur], Af[0]); SCHED(); \
      Af[3] = RDA(so, 3); SCHED(); MMA4(1, Bf[cur], Af[1]); SCHED(); \
      Af[0] = RDA(so, 4); SCHED(); MMA4(2, Bf[cur], Af[2]); SCHED(); \
      Af[1] = RDA(so, 5); SCHED(); MMA4(3, Bf[cur], Af[3]); SCHED(); \
      Af[2] = RDA(so, 6); Af[3] = RDA(so, 7); SCHED(); MMA4(4, Bf[cur], Af[0]); SCHED(); \
      MMA4(5, Bf[cur], Af[1]); SCHED(); \
      asm volatile("s_waitcnt lgkmcnt(0)" ::: "memory"); \
      BAR(); SCHED(); \
      Bf[nxt][0] = RDB(sb, 0); Bf[nxt][1] = RDB(sb, 1); Af[0] = RDA(sa, 0); SCHED(); \
      MMA4(6, Bf[cur], Af[2]); SCHED(); \
      Bf[nxt][2] = RDB(sb, 2); Bf[nxt][3] = RDB(sb, 3); Af[1] = RDA(sa, 1); SCHED(); \
      MMA4(7, Bf[cur], Af[3]); SCHED(); \
      so = sn; } while (0)
    static_assert(g_off(0) == (NE % NSLOT) * SLOT, "granule 0 must sit in the ring slot of step NE");
    for (int kt = 0; kt < NE; kt += 2) {
      KSTEP(0, 1, 0u, 0u);
      KSTEP(1, 0, (kt == NE - 2) ? wm * 12288u : 0u, (kt == NE - 2) ? gb_off(0) - (g_off(0) + A_BYTES) : 0u);
    }
    unsigned bq[4], ga0 = aoff0 - wm * 12288u + g_off(1), ga1 = ga0 ^ 64u;
#pragma unroll
    for (int nn = 0; nn < 4; ++nn) { bq[nn] = boff[nn] + (gb_off(1) - A_BYTES); asm volatile("" : "+v"(bq[nn])); }
    asm volatile("" : "+v"(ga0), "+v"(ga1));
    const unsigned d0 = r * 512 + wn * 256 + ((g ^ r) * 16);
#define GRDA(mm) mk8(*(const LAS v4i*)(lds + ga0 + (mm) * 2048), *(const LAS v4i*)(lds + ga1 + (mm) * 2048))
#define GRDB2(nn, r_) __builtin_amdgcn_ds_read_tr8_b64_v2i32((LAS v2i*)(lds + bq[nn] + ((r_) & 1) * 1024 + ((r_) >> 1) * 8192))
#define GRDB(nn) ({ v2i b0 = GRDB2(nn, 0), b1 = GRDB2(nn, 1), b2 = GRDB2(nn, 2), b3 = GRDB2(nn, 3); \
      v4i lo = __builtin_shufflevector(b0, b1, 0, 1, 2, 3), hi = __builtin_shufflevector(b2, b3, 0, 1, 2, 3); mk8(lo, hi); })
#define MM1(m, nn, Bc, Ac) do { acc[m][nn] = __builtin_amdgcn_mfma_scale_f32_16x16x128_f8f6f4(Bc[nn], Ac, acc[m][nn], 1, 0, 0, 0, 0, 0); SCHED(); } while (0)
#define PSTEP(s) do { \
      WAIT_LGKM0(); BARM(); SCHED(); \
      MM1(((s) >> 2) * 2, 0, Bf[(s) & 1], Af[((s) & 1) * 2]); \
      if ((s) < 15) { Bf[((s) + 1) & 1][0] = GRDB(0); SCHED(); } \
      MM1(((s) >> 2) * 2, 1, Bf[(s) & 1], Af[((s) & 1) * 2]); \
      if ((s) < 15) { Af[(((s) + 1) & 1) * 2] = GRDA(0); SCHED(); } \
      MM1(((s) >> 2) * 2, 2, Bf[(s) & 1], Af[((s) & 1) * 2]); \
      if ((s) < 15) { Bf[((s) + 1) & 1][1] = GRDB(1); SCHED(); } \
      MM1(((s) >> 2) * 2, 3, Bf[(s) & 1], Af[((s) & 1) * 2]); \
      if ((s) < 15) { Bf[((s) + 1) & 1][2] = GRDB(2); SCHED(); } \
      MM1(((s) >> 2) * 2 + 1, 0, Bf[(s) & 1], Af[((s) & 1) * 2 + 1]); \
      if ((s) < 15) { Bf[((s) + 1) & 1][3] = GRDB(3); SCHED(); } \
      MM1(((s) >> 2) * 2 + 1, 1, Bf[(s) & 1], Af[((s) & 1) * 2 + 1]); \
      if ((s) < 15) { Af[(((s) + 1) & 1) * 2 + 1] = GRDA(1); SCHED(); } \
      MM1(((s) >> 2) * 2 + 1, 2, Bf[(s) & 1], Af[((s) & 1) * 2 + 1]); \
      if ((s) < 15) { _Pragma("unroll") for (int nn = 0; nn < 4; ++nn) { bq[nn] += gb_off(((s) + 2) & 3) - gb_off(((s) + 1) & 3); asm volatile("" : "+v"(bq[nn])); } \
        ga0 += g_off((s) < 14 ? (s) + 2 : 15) - g_off((s) + 1); ga1 += g_off((s) < 14 ? (s) + 2 : 15) - g_off((s) + 1); asm volatile("" : "+v"(ga0), "+v"(ga1)); SCHED(); } \
      MM1(((s) >> 2) * 2 + 1, 3, Bf[(s) & 1], Af[((s) & 1) * 2 + 1]); \
      if (((s) & 3) == 3) { unsigned db = d0 + (wm ? st_off((s) >> 2, 1) : st_off((s) >> 2, 0)); asm volatile("" : "+v"(db)); \
        _Pragma("unroll") for (int nn = 0; nn < 4; ++nn) _Pragma("unroll") for (int mm = 0; mm < 2; ++mm) \
          *(LAS v4f*)(lds + (db ^ (nn * 64)) + mm * 8192) = acc[((s) >> 2) * 2 + mm][nn]; SCHED(); } \
      } while (0)
    PSTEP(0); PSTEP(1); PSTEP(2); PSTEP(3);
    PSTEP(4); PSTEP(5); PSTEP(6); PSTEP(7);
    PSTEP(8); PSTEP(9); PSTEP(10); PSTEP(11);
    PSTEP(12); PSTEP(13); PSTEP(14);
    PSTEP(15);
    WAIT_LGKM0(); BARM();
  } else {
    const int hw = wid - 4;
    const bool local = (src == a.rank);
    const int pos0 = PBASE + (tn & 7);
    unsigned* const tmo = (unsigned*)(myws + WS_TMO);
    const __amdgpu_buffer_rsrc_t rsC = __builtin_amdgcn_make_buffer_rsrc((void*)(a.out + (size_t)(tm * BM) * NS + tn * BN), 0, BM * NS * 4, 0x00020000);
    const unsigned vG = (lane >> 5) * (NS * 4) + (lane & 31) * 16;
    const unsigned vL = (lane >> 5) * 512 + (((lane & 31) ^ (lane >> 5)) * 16);
    const float sc = a.sx[0] * a.sw[0];
    float s;
    v4f d[16];
    const __amdgpu_buffer_rsrc_t rsB = __builtin_amdgcn_make_buffer_rsrc((void*)(a.W + a.rank * NS + tn * BN), 0, (unsigned)((size_t)K * NW - (a.rank * NS + tn * BN)), 0x00020000);
    const int lq = lane >> 3, lc = lane & 7;
    const unsigned vB0 = lq * NW + ((lc ^ (lq >> 1)) * 16);
    const unsigned vB1 = lq * NW + ((lc ^ ((lq >> 1) | 4)) * 16);
#define DMA_B(boffs, j, kt) __builtin_amdgcn_raw_ptr_buffer_load_lds(rsB, (LAS void*)(lds + (boffs) + (j) * 1024), 16, (((j) >> 1) & 1) ? vB1 : vB0, ((kt) * BK + (j) * 8) * NW, 0, 0)
#define ISSUE_B(j, j0, j1) do { _Pragma("unroll") for (int jj = (j0); jj < (j1); ++jj) DMA_B(gb_off(j), jj, NE + (j)); } while (0)
#define ORD(pb, sw, i) (*(const LAS v4f*)(lds + st_off(pb, sw) + (i) * 1024 + (vL ^ (((2 * (i)) & 15) * 16))))
#define OST(pb, sw, i, v) __builtin_amdgcn_raw_buffer_store_b128(__builtin_bit_cast(u32x4, v), rsC, vG, ((sw) * 128 + (pb) * 32 + 2 * (i)) * (NS * 4), 16)
#define STNOP() do { SCHED(); asm volatile("s_nop 1"); SCHED(); } while (0)
#define DRAIN(pb, sw, LASTB) do { \
      _Pragma("unroll") for (int i = 0; i < 16; ++i) d[i] = ORD(pb, sw, i); \
      _Pragma("unroll") for (int i = 0; i < 16; ++i) { d[i] = d[i] * s; asm volatile("" : "+v"(d[i])); } \
      SCHED(); \
      _Pragma("unroll") for (int i = 0; i < 4; ++i) OST(pb, sw, i, d[i]); \
      STNOP(); WAIT_LGKM0(); BARM(); \
      _Pragma("unroll") for (int i = 4; i < 8; ++i) OST(pb, sw, i, d[i]); \
      STNOP(); BARM(); \
      _Pragma("unroll") for (int i = 8; i < 12; ++i) OST(pb, sw, i, d[i]); \
      STNOP(); BARM(); \
      _Pragma("unroll") for (int i = 12; i < 16; ++i) OST(pb, sw, i, d[i]); \
      STNOP(); \
      if (LASTB) BARM(); } while (0)
    if (hw == 0) {
      const unsigned long long t0 = __builtin_amdgcn_s_memrealtime();
#define WAIT_FLAGS(f) do { for (;;) { \
          const unsigned v_ = __hip_atomic_load(f, __ATOMIC_RELAXED, __HIP_MEMORY_SCOPE_SYSTEM); \
          const unsigned tv_ = __hip_atomic_load(tmo, __ATOMIC_RELAXED, __HIP_MEMORY_SCOPE_AGENT); \
          if (__all(v_ >= call)) break; \
          if (tv_ == call || __builtin_amdgcn_s_memrealtime() - t0 > DEADLINE_TICKS) { \
            if (lane == 0) __hip_atomic_store(tmo, call, __ATOMIC_RELAXED, __HIP_MEMORY_SCOPE_AGENT); \
            asm volatile("s_waitcnt vmcnt(0)" ::: "memory"); \
            break; \
          } } } while (0)
#define WAIT_FLAGS2(f) do { for (;;) { \
          const unsigned v_ = __hip_atomic_load(f, __ATOMIC_RELAXED, __HIP_MEMORY_SCOPE_SYSTEM); \
          const unsigned w_ = __hip_atomic_load(f + 64, __ATOMIC_RELAXED, __HIP_MEMORY_SCOPE_SYSTEM); \
          const unsigned tv_ = __hip_atomic_load(tmo, __ATOMIC_RELAXED, __HIP_MEMORY_SCOPE_AGENT); \
          if (__all(v_ >= call && w_ >= call)) break; \
          if (tv_ == call || __builtin_amdgcn_s_memrealtime() - t0 > DEADLINE_TICKS) { \
            if (lane == 0) __hip_atomic_store(tmo, call, __ATOMIC_RELAXED, __HIP_MEMORY_SCOPE_AGENT); \
            asm volatile("s_waitcnt vmcnt(0)" ::: "memory"); \
            break; \
          } } } while (0)
      const unsigned* fE = (const unsigned*)(myws + WS_FLAGE) + src * 128 + lane;
      const unsigned* fP = (const unsigned*)(myws + WS_FLAGP) + src * 64 + half * 32 + (lane & 31);
      if (!local) {
        WAIT_FLAGS2(fE);
      }
      BAR();
      if (lane == 0) ticket = __hip_atomic_fetch_add((unsigned*)(myws + WS_TICK), 1u, __ATOMIC_RELAXED, __HIP_MEMORY_SCOPE_AGENT);
      BAR();
      const int JW = pos0 - 3;
      int js = 0;
      if (!local && NPUSH > 0) {
        bool seen = false;
        while (js + 2 <= JW) {
          const unsigned vP = __hip_atomic_load(fP, __ATOMIC_RELAXED, __HIP_MEMORY_SCOPE_SYSTEM);
          BAR();
          seen = __all(vP >= call);
          BAR();
          js += 2;
          if (seen) break;
        }
        if (!seen) {
          for (; js < JW; ++js) BAR();
          WAIT_FLAGS(fP);
        }
      }
      for (; js < NE - 3; ++js) BAR();
      ISSUE_B(0, 0, 16);
      BAR(); BAR();
      ISSUE_B(3, 0, 8);
      unsigned tv2 = __hip_atomic_load(tmo, __ATOMIC_RELAXED, __HIP_MEMORY_SCOPE_AGENT);
      WAIT_VM(9); BAR();
      BAR(); BAR();
      WAIT_VM(0); BAR();
      asm volatile("" : "+v"(tv2));
      s = (tv2 == call) ? __builtin_nanf("") : sc;
      BAR(); BARM();
      DRAIN(0, 0, true); DRAIN(1, 0, true); DRAIN(2, 0, false);
    } else {
      const unsigned char* abase = local ? (a.x + (size_t)half * BM * K) : (a.wsr[xo] + WS_EXP + par * SHARD + (size_t)half * (SHARD / 2));
      const unsigned rowstride = local ? K : BK;
      const unsigned kstride = local ? BK : A_BYTES;
      const __amdgpu_buffer_rsrc_t rsA = __builtin_amdgcn_make_buffer_rsrc((void*)abase, 0, BM * K, 0x00020000);
      const __amdgpu_buffer_rsrc_t rsS = __builtin_amdgcn_make_buffer_rsrc((void*)(myws + WS_STG + (size_t)(par * NR + src) * STG_SRC + (size_t)half * 4 * A_BYTES), 0, 4 * A_BYTES, 0x00020000);
      const unsigned vA0 = lq * rowstride + ((lc ^ (lane >> 4)) * 16);
      const unsigned vA1 = lq * rowstride + ((lc ^ (4 + (lane >> 4))) * 16);
      int nj = 0, nkt = local ? 0 : NPUSH, npi = 0, npp = (local || NPUSH == 0) ? 1000 : pos0;
      unsigned islot = 0;
#define DMA_A(rs, slotoff, i, so) __builtin_amdgcn_raw_ptr_buffer_load_lds(rs, (LAS void*)(lds + (slotoff) + (i) * 1024), 16, ((i) & 1) ? vA1 : vA0, (so) + (i) * 8 * rowstride, 0, 0)
#define ISSUE() do { \
        int kt_; bool ps_ = false; \
        if (nj == npp) { kt_ = npi; ps_ = true; ++npi; npp = (npi < NPUSH) ? npp + PSTR : 1000; } else { kt_ = nkt; ++nkt; } \
        if (hw == 3) { _Pragma("unroll") for (int j = 0; j < 16; ++j) DMA_B(islot + A_BYTES, j, kt_); } \
        else { \
          const unsigned lo_ = islot + (hw - 1) * 16384, so_ = kt_ * kstride + (hw - 1) * 128 * rowstride; \
          if (ps_) { _Pragma("unroll") for (int i = 0; i < 16; ++i) DMA_A(rsS, lo_, i, so_); } \
          else { _Pragma("unroll") for (int i = 0; i < 16; ++i) DMA_A(rsA, lo_, i, so_); } \
        } \
        ++nj; islot += SLOT; if (islot == NSLOT * SLOT) islot = 0; } while (0)
#define ISSUE_G(gg) do { \
          const unsigned lo_ = g_off(gg) + (hw - 1) * 4096; \
          const unsigned so_ = local ? (NE + ((gg) & 3)) * BK + ((hw - 1) * 128 + ((gg) >> 2) * 32) * K : NE * A_BYTES + ((gg) * 2 + (hw - 1)) * 4096; \
          _Pragma("unroll") for (int i = 0; i < 4; ++i) DMA_A(rsA, lo_, i, so_); } while (0)
      if (hw == 3) { ISSUE(); ISSUE(); ISSUE(); }
      BAR();
      if (hw != 3) { ISSUE(); ISSUE(); ISSUE(); }
      WAIT_VM(32);
      BAR();
      for (int js = 0; js < NE - 3; ++js) {
        WAIT_VM(16);
        BAR();
        ISSUE();
      }
      if (hw != 3) {
#define GSTEP(sidx) do { WAIT_VM((sidx) == -3 ? 16 : g_allow(sidx)); BAR(); \
          _Pragma("unroll") for (int gg = 0; gg < 16; ++gg) if (g_issue(gg) == (sidx)) ISSUE_G(gg); } while (0)
        GSTEP(-3); GSTEP(-2); GSTEP(-1);
        GSTEP(0); GSTEP(1); GSTEP(2); GSTEP(3); GSTEP(4); GSTEP(5); GSTEP(6); GSTEP(7);
        GSTEP(8); GSTEP(9); GSTEP(10); GSTEP(11); GSTEP(12); GSTEP(13); GSTEP(14);
        BAR();
      } else {
        WAIT_VM(16); BAR(); ISSUE_B(1, 0, 16); ISSUE_B(2, 0, 16);
        WAIT_VM(32); BAR(); ISSUE_B(3, 8, 16);
        unsigned tv2 = __hip_atomic_load(tmo, __ATOMIC_RELAXED, __HIP_MEMORY_SCOPE_AGENT);
        BAR();
        WAIT_VM(25); BAR();
        WAIT_VM(9); BAR();
        WAIT_VM(0); BAR();
        asm volatile("" : "+v"(tv2));
        s = (tv2 == call) ? __builtin_nanf("") : sc;
        BAR(); BARM();
        DRAIN(0, 1, true); DRAIN(1, 1, true); DRAIN(2, 1, false);
      }
    }
    BARM();
    if (hw == 0 || hw == 3) {
      const int sw = hw & 1;
      const unsigned lb = sw ? st_off(3, 1) : st_off(3, 0);
      const unsigned gbo = (sw * 128 + 3 * 32) * (NS * 4);
#pragma unroll
      for (int i = 0; i < 16; ++i) d[i] = *(const LAS v4f*)(lds + lb + i * 1024 + (vL ^ (((2 * i) & 15) * 16)));
#pragma unroll
      for (int i = 0; i < 16; ++i) { d[i] = d[i] * s; asm volatile("" : "+v"(d[i])); }
      SCHED();
#pragma unroll
      for (int i = 0; i < 16; ++i) __builtin_amdgcn_raw_buffer_store_b128(__builtin_bit_cast(u32x4, d[i]), rsC, vG, gbo + 2 * i * (NS * 4), 16);
      STNOP();
    }
  }
  if (tid == 256 && (ticket % NWG) == NWG - 1)
    __hip_atomic_store((unsigned*)(myws + WS_CTR), call, __ATOMIC_RELAXED, __HIP_MEMORY_SCOPE_AGENT);
}

}

extern "C" void kernel_launch(int rank, int nranks, ncclComm_t comm,
                              hipStream_t stream,
                              void* const* d_in, const int* in_sizes, int n_in,
                              void* d_out, int out_size,
                              const int* in_dtypes, int out_dtype,
                              void* const* d_out_all,
                              void* const* d_ws_all, size_t ws_bytes) {
  if (nranks != NR || n_in < 4) { fprintf(stderr, "unsupported configuration\n"); return; }
  Args a;
  a.x = (const unsigned char*)d_in[0];
  a.W = (const unsigned char*)d_in[1];
  a.sx = (const float*)d_in[2];
  a.sw = (const float*)d_in[3];
  a.out = (float*)d_out;
  for (int p = 0; p < NR; ++p) a.wsr[p] = (unsigned char*)d_ws_all[(rank + p) & 7];
  a.rank = rank;
  const unsigned perm_tab[NR] = {0x75416320u, 0x75216430u, 0x75326410u, 0x64217530u, 0x76425310u, 0x65317420u, 0x76315420u, 0x74316520u};
  a.perm = perm_tab[rank & 7];
  hipLaunchKernelGGL(fused_gemm, dim3(NWG), dim3(512), 0, stream, a);
  HIP_CHECK(hipGetLastError());
}
```
